```python
import numpy as np

kernel_name = "gpu_gemm_a2a_m4096_k4096_n4096_f32_silu"


ABSMAX_RTOL = 0.001

COMPUTE_DTYPE = "float32"
EPILOGUE = "silu"
IN_QUANT = None
OUT_QUANT = None
IN_DTYPES = {"x": "float32", "w_mat": "float32"}
WORLD_SIZE = 8

def setup_inputs():

    x = np.random.randn(4096, 4096).astype(np.float32)
    w_mat = (np.random.randn(4096, 4096) / np.sqrt(4096)).astype(np.float32)
    return {"x": x, "w_mat": w_mat}

def _epilogue(y):
    return (y / (1.0 + np.exp(-np.clip(y, -60.0, 60.0)))).astype(np.float32)

def reference(x, w_mat):
    y = (x @ w_mat).astype(np.float32)
    return _epilogue(y)
```

```cpp
#include <hip/hip_runtime.h>
#include <rccl/rccl.h>
#include <cstdio>

typedef float f32x4 __attribute__((ext_vector_type(4)));
typedef float f32x16 __attribute__((ext_vector_type(16)));
typedef _Float16 f16x8 __attribute__((ext_vector_type(8)));
typedef _Float16 f16x4 __attribute__((ext_vector_type(4)));
typedef unsigned int u32x4 __attribute__((ext_vector_type(4)));

constexpr int NR = 8;
constexpr int ML = 512;
constexpr int KK = 4096, NN = 4096;
constexpr int NS = 512;
constexpr unsigned long long DEADLINE = 20000000ull;

constexpr size_t WS_TICKET = 0;
constexpr size_t WS_TMO = 128;
constexpr size_t WS_DONE = 256;
constexpr size_t WS_FLAG = 1024;

struct Params {
  const float* x; const float* w;
  float* out[NR];
  char* ws[NR];
  int rank;
};

__device__ __forceinline__ float silu(float y) {
  float c = fminf(fmaxf(y, -60.f), 60.f);
  return y / (1.f + __expf(-c));
}

constexpr int BM = 128, BN = 64, BK = 32;
constexpr int LDA = BK + 8;

__global__ void __launch_bounds__(256, 1) fused(Params p) {
  __shared__ __attribute__((aligned(16))) _Float16 sA[BM * LDA];
  __shared__ __attribute__((aligned(16))) _Float16 sB[BN * LDA];
  __shared__ unsigned s_call;
  const int tid = threadIdx.x, lane = tid & 63, wid = tid >> 6;
  char* myws = p.ws[p.rank];
  if (tid == 0) {
    unsigned t = __hip_atomic_fetch_add((unsigned*)(myws + WS_TICKET), 1u, __ATOMIC_RELAXED, __HIP_MEMORY_SCOPE_AGENT);
    s_call = t / gridDim.x + 1;
  }
  const int g = blockIdx.x & 7, idx = blockIdx.x >> 3;
  const int dest = (p.rank + 1 + g) & 7;
  const int mt = idx & 3, nt = idx >> 2;
  const int m0 = mt * BM, n0 = dest * NS + nt * BN;
  const int wm = wid & 1, wn = wid >> 1;
  f32x16 acc[2] = {};
  for (int k0 = 0; k0 < KK; k0 += BK) {
#pragma unroll
    for (int i = 0; i < 4; ++i) {
      int e = tid + i * 256; int r = e >> 3, c = (e & 7) * 4;
      f32x4 v = *(const f32x4*)(p.x + (size_t)(m0 + r) * KK + k0 + c);
      f16x4 h = {(_Float16)v[0], (_Float16)v[1], (_Float16)v[2], (_Float16)v[3]};
      *(f16x4*)(sA + r * LDA + c) = h;
    }
#pragma unroll
    for (int i = 0; i < 2; ++i) {
      int e = tid + i * 256; int r = e >> 4, c = (e & 15) * 4;
      f32x4 v = *(const f32x4*)(p.w + (size_t)(k0 + r) * NN + n0 + c);
#pragma unroll
      for (int j = 0; j < 4; ++j) sB[(c + j) * LDA + r] = (_Float16)v[j];
    }
    __syncthreads();
#pragma unroll
    for (int ks = 0; ks < BK; ks += 16) {
      f16x8 b = *(const f16x8*)(sB + (wn * 32 + (lane & 31)) * LDA + ks + (lane >> 5) * 8);
#pragma unroll
      for (int m = 0; m < 2; ++m) {
        f16x8 a = *(const f16x8*)(sA + (wm * 64 + m * 32 + (lane & 31)) * LDA + ks + (lane >> 5) * 8);
        acc[m] = __builtin_amdgcn_mfma_f32_32x32x16_f16(a, b, acc[m], 0, 0, 0);
      }
    }
    __syncthreads();
  }
  const unsigned call = s_call;
  float* ob = p.out[dest];
  __amdgpu_buffer_rsrc_t rs = __builtin_amdgcn_make_buffer_rsrc(ob, 0, (int)(4096u * NS * 4u), 0x00020000);
#pragma unroll
  for (int m = 0; m < 2; ++m)
#pragma unroll
    for (int r = 0; r < 16; ++r) {
      int row = m0 + wm * 64 + m * 32 + (r & 3) + 8 * (r >> 2) + 4 * (lane >> 5);
      int col = nt * BN + wn * 32 + (lane & 31);
      unsigned off = ((unsigned)(p.rank * ML + row) * NS + col) * 4u;
      __builtin_amdgcn_raw_buffer_store_b32(__builtin_bit_cast(unsigned, silu(acc[m][r])), rs, off, 0, 17);
    }
  asm volatile("s_waitcnt vmcnt(0)" ::: "memory");
  __syncthreads();
  if (wid == 0) {
    int last = 0;
    if (lane == 0) {
      unsigned t = __hip_atomic_fetch_add((unsigned*)(myws + WS_DONE), 1u, __ATOMIC_RELEASE, __HIP_MEMORY_SCOPE_AGENT);
      last = (t + 1 == call * gridDim.x);
    }
    last = __shfl(last, 0);
    if (last) {
      __builtin_amdgcn_fence(__ATOMIC_ACQUIRE, "agent");
      __builtin_amdgcn_fence(__ATOMIC_RELEASE, "");
      asm volatile("s_waitcnt vmcnt(0)" ::: "memory");
      if (lane < NR && lane != p.rank) {
        __hip_atomic_store((unsigned*)(p.ws[lane] + WS_FLAG + p.rank * 128), call, __ATOMIC_RELAXED, __HIP_MEMORY_SCOPE_SYSTEM);
      }
      bool bad = false;
      if (lane < NR && lane != p.rank) {
        unsigned* f = (unsigned*)(myws + WS_FLAG + lane * 128);
        unsigned* tmo = (unsigned*)(myws + WS_TMO);
        unsigned long long t0 = __builtin_amdgcn_s_memrealtime();
        while (__hip_atomic_load(f, __ATOMIC_RELAXED, __HIP_MEMORY_SCOPE_SYSTEM) < call) {
          if (__hip_atomic_load(tmo, __ATOMIC_RELAXED, __HIP_MEMORY_SCOPE_AGENT) == call ||
              __builtin_amdgcn_s_memrealtime() - t0 > DEADLINE) {
            __hip_atomic_store(tmo, call, __ATOMIC_RELAXED, __HIP_MEMORY_SCOPE_AGENT);
            bad = true;
            break;
          }
          __builtin_amdgcn_s_sleep(2);
        }
      }
      __builtin_amdgcn_fence(__ATOMIC_ACQUIRE, "");
      if (bad) p.out[p.rank][(size_t)p.rank * ML * NS] = __builtin_nanf("");
    }
  }
}

extern "C" void kernel_launch(int rank, int nranks, ncclComm_t comm, hipStream_t stream,
                              void* const* d_in, const int* in_sizes, int n_in,
                              void* d_out, int out_size, const int* in_dtypes, int out_dtype,
                              void* const* d_out_all, void* const* d_ws_all, size_t ws_bytes) {
  if (nranks != NR) { fprintf(stderr, "unsupported nranks\n"); return; }
  Params p;
  p.x = (const float*)d_in[0];
  p.w = (const float*)d_in[1];
  for (int i = 0; i < NR; ++i) { p.out[i] = (float*)d_out_all[i]; p.ws[i] = (char*)d_ws_all[i]; }
  p.rank = rank;
  fused<<<256, 256, 0, stream>>>(p);
}
```

```cpp
#include <hip/hip_runtime.h>
#include <rccl/rccl.h>
#include <cstdio>
#include <type_traits>

typedef float f32x4 __attribute__((ext_vector_type(4)));
typedef _Float16 f16x8 __attribute__((ext_vector_type(8)));
typedef _Float16 f16x2 __attribute__((ext_vector_type(2)));
typedef float f32x2 __attribute__((ext_vector_type(2)));
typedef unsigned int u32x4 __attribute__((ext_vector_type(4)));
typedef unsigned int u32x2 __attribute__((ext_vector_type(2)));

constexpr int NR = 8;
constexpr int ML = 512;
constexpr int KK = 4096, NN = 4096;
constexpr int NS = 512;
constexpr int ROUNDS = 16;
constexpr int GRID = 256;
constexpr unsigned long long DEADLINE = 20000000ull;
constexpr unsigned LDS_SPIN_CAP = 1u << 24;

constexpr size_t WS_TICKET = 0;
constexpr size_t WS_TMO = 1024;
constexpr size_t WS_CNT = 4096;
constexpr size_t WS_SLAB = 1u << 20;
constexpr size_t SLAB_BYTES = (size_t)128 * 8 * 4 * 3 * 2048;
constexpr size_t WS_NEED = WS_SLAB + SLAB_BYTES;

constexpr int RING_BYTES = 65536;
constexpr int LDS_FULL = 2 * RING_BYTES;
constexpr int LDS_CONS = LDS_FULL + 16;
constexpr int LDS_CALL = LDS_FULL + 32;
constexpr int LDS_DONE = LDS_FULL + 36;
constexpr int LDS_AISS = LDS_FULL + 40;
constexpr int LDS_ARDY = LDS_FULL + 44;
constexpr int LDS_BYTES = LDS_FULL + 64;

struct Params {
  const float* x;
  const float* w;
  float* out[NR];
  char* ws[NR];
  int rank;
};

#define LD_NB 2
#define W_AUX 0
constexpr int AUX_SC1 = 16;
constexpr int AUX_SYS = 17;

__device__ __forceinline__ float silu(float y) {
  float c = fminf(fmaxf(y, -60.f), 60.f);
  return y * __builtin_amdgcn_rcpf(1.f + __expf(-c));
}

__device__ __forceinline__ unsigned pack2(float a, float b) {
  f32x2 v = {a, b};
  f16x2 h = __builtin_convertvector(v, f16x2);
  return __builtin_bit_cast(unsigned, h);
}

typedef __attribute__((address_space(3))) unsigned lds_u32;
typedef __attribute__((address_space(3))) u32x4 lds_u32x4;

__device__ __forceinline__ unsigned lds_ld(const char* smem, int off) {
  return __hip_atomic_load((const unsigned*)(smem + off), __ATOMIC_RELAXED, __HIP_MEMORY_SCOPE_WORKGROUP);
}
__device__ __forceinline__ void lds_st(char* smem, int off, unsigned v) {
  __hip_atomic_store((unsigned*)(smem + off), v, __ATOMIC_RELAXED, __HIP_MEMORY_SCOPE_WORKGROUP);
}

__device__ __forceinline__ void lds_wait4(const char* smem, int off, unsigned need) {
#pragma unroll 1
  for (unsigned spin = 0; spin < LDS_SPIN_CAP; ++spin) {
    const u32x4 v = *(const volatile lds_u32x4*)(smem + off);
    if (min(min(v[0], v[1]), min(v[2], v[3])) >= need) break;
    __builtin_amdgcn_s_sleep(1);
  }
  asm volatile("" ::: "memory");
}

__device__ __forceinline__ void strip_map(int s, int rank, int& dest, int& dcol) {
  if (s < 112) {
    int sd = (s * 147) >> 10;
    dest = (rank + 1 + (s - sd * 7)) & 7;
    dcol = sd * 32;
  } else {
    dest = rank;
    dcol = (s - 112) * 32;
  }
}

#define RSTAMP(i) do {} while (0)
#define STAMP(i) do {} while (0)
#define STAMP_FLUSH() do {} while (0)

__device__ __forceinline__ void lds_wait1(const char* smem, int off, unsigned need) {
#pragma unroll 1
  for (unsigned spin = 0; spin < LDS_SPIN_CAP; ++spin) {
    if (lds_ld(smem, off) >= need) break;
    __builtin_amdgcn_s_sleep(1);
  }
  asm volatile("" ::: "memory");
}
__device__ __forceinline__ unsigned lds_inc(char* smem, int off) {
  return __hip_atomic_fetch_add((unsigned*)(smem + off), 1u, __ATOMIC_RELAXED, __HIP_MEMORY_SCOPE_WORKGROUP);
}

__global__ void __launch_bounds__(512, 2) fused(Params p) {
  __shared__ __attribute__((aligned(16))) char smem[LDS_BYTES];
  const int tid = threadIdx.x, lane = tid & 63;
  const int wid = __builtin_amdgcn_readfirstlane(tid >> 6);
  char* const myws = p.ws[p.rank];
  const int bid = blockIdx.x;
  const int g8 = bid & 7, idx = bid >> 3;
  const int nh = g8 & 1, kq = g8 >> 1;
  const int mi = idx & 7, ng = idx >> 3;
  const int rot = 2 * mi;

  if (tid < 16) *(unsigned*)(smem + LDS_FULL + 4 * tid) = 0u;
  __syncthreads();

  STAMP(0);

  if (wid >= 4) {
    const int q = wid - 4;
    constexpr int LAST_STAMP_IDX = 16; (void)LAST_STAMP_IDX;
    const __amdgpu_buffer_rsrc_t rw =
        __builtin_amdgcn_make_buffer_rsrc((void*)p.w, 0, (int)((unsigned)KK * NN * 4u), 0x00020000);
    const int G = lane >> 3, c = lane & 7;
    const int sl = G >> 2, g = G & 3;
    unsigned voff[8];
#pragma unroll
    for (int jj = 0; jj < 8; ++jj) voff[jj] = (unsigned)(sl * 32 + 8 * g + jj) * (NN * 4u) + (unsigned)c * 16u;
    unsigned rowoff[4];
#pragma unroll
    for (int c4 = 0; c4 < 4; ++c4) rowoff[c4] = (unsigned)(kq * 1024 + ((4 * q + c4 + rot) & 15) * 64) * (NN * 4u);
    unsigned coltab;
    {
      int dest, dcol;
      strip_map((lane & 15) * 8 + nh * 4 + ng, p.rank, dest, dcol);
      coltab = (unsigned)(dest * NS + dcol) * 4u;
    }
    const unsigned wbase = (unsigned)(g * 256 + sl * 2048 + c * 16);
    f32x4 buf[LD_NB][8];

    auto issue = [&](unsigned col, int c4, int slot) {
      const unsigned base = rowoff[c4] + col;
#pragma unroll
      for (int jj = 0; jj < 8; ++jj)
        buf[slot][jj] = __builtin_bit_cast(f32x4, __builtin_amdgcn_raw_buffer_load_b128(rw, voff[jj], base, W_AUX));
    };
    {
      const unsigned col0 = __builtin_amdgcn_readlane(coltab, 0);
#pragma unroll
      for (int e = 0; e < LD_NB; ++e) {
        issue(col0, e, e);
        __builtin_amdgcn_sched_barrier(0);
      }
    }
    if (q == 3 && lane == 0) {
      const unsigned ticket = __hip_atomic_fetch_add((unsigned*)(myws + WS_TICKET + g8 * 128), 1u, __ATOMIC_RELAXED,
                                                     __HIP_MEMORY_SCOPE_AGENT);
      lds_st(smem, LDS_CALL, ticket / (GRID / 8) + 1u);
    }

    auto round_step = [&](int j, auto more_tag) {
      constexpr bool MORE = decltype(more_tag)::value;
      char* ring = smem + (j & 1) * RING_BYTES + q * 16384;
      const unsigned col = __builtin_amdgcn_readlane(coltab, j);
      const unsigned coln = __builtin_amdgcn_readlane(coltab, (j + 1) & 15);
#pragma unroll
      for (int c4 = 0; c4 < 4; ++c4) {
        const int slot = c4 % LD_NB;
        u32x4 pk[4];
#pragma unroll
        for (int i = 0; i < 4; ++i) {
#pragma unroll
          for (int h = 0; h < 4; ++h) pk[i][h] = pack2(buf[slot][2 * h][i], buf[slot][2 * h + 1][i]);
        }
#pragma unroll
        for (int i = 0; i < 4; ++i) asm volatile("" : "+v"(pk[i]));
        if (c4 + LD_NB < 4) issue(col, c4 + LD_NB, slot);
        else if (MORE) {
          if (j == 0 && c4 + LD_NB == 4) lds_wait1(smem, LDS_AISS, 4u);
          issue(coln, c4 + LD_NB - 4, slot);
        }
        __builtin_amdgcn_sched_barrier(0);
        if (c4 == 0) {
          if (j >= 2) lds_wait4(smem, LDS_CONS, (unsigned)(j - 1));
          else if (j == 1) lds_wait1(smem, LDS_ARDY, 4u);
        }
#pragma unroll
        for (int i = 0; i < 4; ++i)
          *(u32x4*)(ring + c4 * 4096 + (i & 1) * 1024 + (i >> 1) * 128 + wbase) = pk[i];
      }
      asm volatile("s_waitcnt lgkmcnt(0)" ::: "memory");
      if (lane == 0) lds_st(smem, LDS_FULL + 4 * q, (unsigned)(j + 1));
      RSTAMP(1 + j);
    };
#pragma unroll 1
    for (int j = 0; j < ROUNDS - 1; ++j) round_step(j, std::true_type{});
    round_step(ROUNDS - 1, std::false_type{});
    STAMP_FLUSH();
    return;
  }

  const int w = wid;
  constexpr int LAST_STAMP_IDX = 17; (void)LAST_STAMP_IDX;
  f16x8 A[32];
  {
    const __amdgpu_buffer_rsrc_t rx =
        __builtin_amdgcn_make_buffer_rsrc((void*)p.x, 0, (int)((unsigned)ML * KK * 4u), 0x00020000);
    char* stage = smem + RING_BYTES + w * 16384;
    f32x4 t[2][16];
    auto lda = [&](int b) {
      const int ac = (b * 4 + (lane >> 4) + rot) & 15;
      const unsigned voff = (unsigned)(kq * 1024 + ac * 64 + (lane & 15) * 4) * 4u;
#pragma unroll
      for (int r = 0; r < 16; ++r)
        t[b & 1][r] = __builtin_bit_cast(f32x4, __builtin_amdgcn_raw_buffer_load_b128(
                                                    rx, voff, (unsigned)(mi * 64 + w * 16 + r) * (KK * 4u), 0));
    };
    auto cva = [&](int b) {
      char* sb = stage + (b & 1) * 8192;
      const int ch = lane >> 1;
#pragma unroll
      for (int r = 0; r < 16; ++r) {
        u32x2 h = {pack2(t[b & 1][r][0], t[b & 1][r][1]), pack2(t[b & 1][r][2], t[b & 1][r][3])};
        *(u32x2*)(sb + r * 512 + (((ch & 16) | ((ch ^ r) & 15)) << 4) + (lane & 1) * 8) = h;
      }
    };
    auto rda = [&](int b) {
      const char* sb = stage + (b & 1) * 8192;
      const int r = lane & 15;
#pragma unroll
      for (int e = 0; e < 8; ++e) {
        const int ch = e * 4 + (lane >> 4);
        A[b * 8 + e] = *(const f16x8*)(sb + r * 512 + (((ch & 16) | ((ch ^ r) & 15)) << 4));
      }
    };
    {
    lda(0); lda(1);
    __builtin_amdgcn_sched_barrier(0);
    cva(0);
    __builtin_amdgcn_sched_barrier(0);
    lda(2);
    __builtin_amdgcn_sched_barrier(0);
    rda(0); cva(1);
    __builtin_amdgcn_sched_barrier(0);
    lda(3);
    if (lane == 0) lds_inc(smem, LDS_AISS);
    __builtin_amdgcn_sched_barrier(0);
    rda(1);
    asm volatile("s_waitcnt lgkmcnt(0)" ::: "memory");
    cva(2);
    __builtin_amdgcn_sched_barrier(0);
    rda(2); cva(3);
    __builtin_amdgcn_sched_barrier(0);
    rda(3);
    asm volatile("s_waitcnt lgkmcnt(0)" ::: "memory");
#pragma unroll
    for (int e = 0; e < 32; ++e) asm volatile("" : "+v"(A[e]));
    if (lane == 0) lds_inc(smem, LDS_ARDY);
    }
  }
  unsigned call = 0;
#pragma unroll 1
  for (unsigned spin = 0; spin < LDS_SPIN_CAP; ++spin) {
    call = lds_ld(smem, LDS_CALL);
    if (call != 0u) break;
    __builtin_amdgcn_s_sleep(1);
  }
  call = __builtin_amdgcn_readfirstlane(call);
  const unsigned tag = call & 1u;
  STAMP(1);

  const __amdgpu_buffer_rsrc_t rs =
      __builtin_amdgcn_make_buffer_rsrc(myws + WS_SLAB, 0, (int)SLAB_BYTES, 0x00020000);
  unsigned* const tmo = (unsigned*)(myws + WS_TMO);

  f32x4 pend0 = {0.f, 0.f, 0.f, 0.f}, pend1 = {0.f, 0.f, 0.f, 0.f};
  u32x4 pr[6];
  unsigned tm = 0;

  auto tile_off = [&](int j) { return (unsigned)(((j * 8 + nh * 4 + ng) * 8 + mi) * 4 + w) * 3u * 2048u; };
  auto gather_issue = [&](int j) {
    const unsigned o = tile_off(j) + (unsigned)lane * 16u;
#pragma unroll
    for (int i = 0; i < 6; ++i) pr[i] = __builtin_amdgcn_raw_buffer_load_b128(rs, o + (unsigned)i * 1024u, 0, AUX_SC1);
    tm = __hip_atomic_load(tmo, __ATOMIC_RELAXED, __HIP_MEMORY_SCOPE_AGENT);
  };
  auto gather_finish = [&](int j) {
    bool bad = false;
    unsigned long long t0 = 0;
    for (int tries = 0;; ++tries) {
      unsigned all1 = ~0u, any1 = 0u;
#pragma unroll
      for (int i = 0; i < 6; ++i)
#pragma unroll
        for (int e = 0; e < 4; ++e) { all1 &= pr[i][e]; any1 |= pr[i][e]; }
      const bool ok = tag ? (all1 & 1u) : !(any1 & 1u);
      if (tm == call) { bad = true; break; }
      if (__all(ok)) break;
      const unsigned long long now = __builtin_amdgcn_s_memrealtime();
      if (tries == 0) t0 = now;
      if (now - t0 > DEADLINE) {
        __hip_atomic_store(tmo, call, __ATOMIC_RELAXED, __HIP_MEMORY_SCOPE_AGENT);
        bad = true;
        break;
      }
      __builtin_amdgcn_s_sleep(1);
      gather_issue(j);
    }
    const float poison = bad ? __builtin_nanf("") : 0.f;
    f32x4 y0, y1;
#pragma unroll
    for (int e = 0; e < 4; ++e) {
      const f32x4 p0 = __builtin_bit_cast(f32x4, pr[0]), p1 = __builtin_bit_cast(f32x4, pr[1]);
      const f32x4 p2 = __builtin_bit_cast(f32x4, pr[2]), p3 = __builtin_bit_cast(f32x4, pr[3]);
      const f32x4 p4 = __builtin_bit_cast(f32x4, pr[4]), p5 = __builtin_bit_cast(f32x4, pr[5]);
      y0[e] = silu(pend0[e] + p0[e] + p2[e] + p4[e]) + poison;
      y1[e] = silu(pend1[e] + p1[e] + p3[e] + p5[e]) + poison;
    }
    const u32x4 u0 = __builtin_bit_cast(u32x4, y0), u1 = __builtin_bit_cast(u32x4, y1);
    int dest, dcol;
    strip_map(j * 8 + nh * 4 + ng, p.rank, dest, dcol);
    const __amdgpu_buffer_rsrc_t ro =
        __builtin_amdgcn_make_buffer_rsrc(p.out[dest], 0, (int)((unsigned)(NR * ML) * NS * 4u), 0x00020000);
    const unsigned row = (unsigned)(p.rank * ML + mi * 64 + w * 16 + 4 * (lane >> 4));
    const unsigned col = (unsigned)(dcol + 4 * (lane & 7) + 2 * ((lane >> 3) & 1));
#pragma unroll
    for (int e = 0; e < 4; ++e) {
      u32x2 v = {u0[e], u1[e]};
      __builtin_amdgcn_raw_buffer_store_b64(v, ro, ((row + e) * NS + col) * 4u, 0, AUX_SYS);
    }
    if (j < 4) STAMP(23); else STAMP(21);
  };
  auto signal_peers = [&]() {
    asm volatile("s_waitcnt vmcnt(0)" ::: "memory");
    unsigned n = 0;
    if (lane == 0) n = lds_inc(smem, LDS_DONE);
    n = __builtin_amdgcn_readfirstlane(n);
    if (n == 3u && lane < NR && lane != p.rank)
      __hip_atomic_fetch_add((unsigned*)(p.ws[lane] + WS_CNT + (p.rank * 8 + g8) * 128), 1u, __ATOMIC_RELAXED,
                             __HIP_MEMORY_SCOPE_SYSTEM);
    STAMP(20);
  };

  for (int j = 0; j < ROUNDS; ++j) {
    f32x4 acc0 = {0.f, 0.f, 0.f, 0.f}, acc1 = {0.f, 0.f, 0.f, 0.f};
    const char* ring = smem + (j & 1) * RING_BYTES + lane * 16;
    lds_wait4(smem, LDS_FULL, (unsigned)(j + 1));
    {
      f16x8 B[2][8];
#pragma unroll
      for (int e = 0; e < 8; ++e) B[0][e] = *(const f16x8*)(ring + e * 1024);
#pragma unroll
      for (int gq = 0; gq < 8; ++gq) {
        if (gq + 1 < 8) {
#pragma unroll
          for (int e = 0; e < 8; ++e) B[(gq + 1) & 1][e] = *(const f16x8*)(ring + ((gq + 1) * 8 + e) * 1024);
        }
        __builtin_amdgcn_sched_barrier(0);
#pragma unroll
        for (int st = 0; st < 4; ++st) {
          acc0 = __builtin_amdgcn_mfma_f32_16x16x32_f16(A[gq * 4 + st], B[gq & 1][2 * st], acc0, 0, 0, 0);
          acc1 = __builtin_amdgcn_mfma_f32_16x16x32_f16(A[gq * 4 + st], B[gq & 1][2 * st + 1], acc1, 0, 0, 0);
        }
        __builtin_amdgcn_sched_barrier(0);
      }
      asm volatile("s_waitcnt lgkmcnt(0)" ::: "memory");
      if (lane == 0) lds_st(smem, LDS_CONS + 4 * w, (unsigned)(j + 1));
    }
    RSTAMP(2 + j);

    const int phase = (j + mi - kq) & 3;
    if (phase == 2 && j >= 2) gather_finish(j - 2);
    if (phase != 0) {
      const int red = (j + mi) & 3;
      const unsigned off = tile_off(j) + (unsigned)((kq - red - 1) & 3) * 2048u + (unsigned)lane * 16u;
      u32x4 v0 = __builtin_bit_cast(u32x4, acc0), v1 = __builtin_bit_cast(u32x4, acc1);
#pragma unroll
      for (int i = 0; i < 4; ++i) { v0[i] = (v0[i] & ~1u) | tag; v1[i] = (v1[i] & ~1u) | tag; }
      __builtin_amdgcn_raw_buffer_store_b128(v0, rs, off, 0, AUX_SC1);
      __builtin_amdgcn_raw_buffer_store_b128(v1, rs, off + 1024u, 0, AUX_SC1);
    } else {
      pend0 = acc0; pend1 = acc1;
    }
    if (phase == 1 && j >= 1) gather_issue(j - 1);
  }
  {
    const int ph = (ROUNDS - 1 + mi - kq) & 3;
    signal_peers();
    if (ph == 0) { gather_issue(ROUNDS - 1); gather_finish(ROUNDS - 1); }
    else if (ph == 1) gather_finish(ROUNDS - 2);
  }

  if (bid == 0 && w == 0) {
    const int src = lane >> 3;
    bool bad = false;
    if (src != p.rank) {
      unsigned* f = (unsigned*)(myws + WS_CNT + lane * 128);
      const unsigned need = call * (GRID / 8);
      const unsigned long long t0 = __builtin_amdgcn_s_memrealtime();
      while (__hip_atomic_load(f, __ATOMIC_RELAXED, __HIP_MEMORY_SCOPE_SYSTEM) < need) {
        if (__hip_atomic_load(tmo, __ATOMIC_RELAXED, __HIP_MEMORY_SCOPE_AGENT) == call ||
            __builtin_amdgcn_s_memrealtime() - t0 > DEADLINE) {
          __hip_atomic_store(tmo, call, __ATOMIC_RELAXED, __HIP_MEMORY_SCOPE_AGENT);
          bad = true;
          break;
        }
        __builtin_amdgcn_s_sleep(1);
      }
    }
    __builtin_amdgcn_fence(__ATOMIC_ACQUIRE, "");
    if (__any(bad)) {
      asm volatile("s_waitcnt vmcnt(0)" ::: "memory");
      if (lane == 0) p.out[p.rank][(size_t)p.rank * ML * NS] = __builtin_nanf("");
    }
    STAMP(22);
  }
  STAMP_FLUSH();
}

extern "C" void kernel_launch(int rank, int nranks, ncclComm_t comm, hipStream_t stream,
                              void* const* d_in, const int* in_sizes, int n_in,
                              void* d_out, int out_size, const int* in_dtypes, int out_dtype,
                              void* const* d_out_all, void* const* d_ws_all, size_t ws_bytes) {
  if (nranks != NR || n_in < 2 || in_sizes[0] != ML * KK || in_sizes[1] != KK * NN || out_size != NR * ML * NS ||
      ws_bytes < WS_NEED || (unsigned)rank >= (unsigned)NR) {
    fprintf(stderr, "kernel_launch: unexpected problem shape\n");
    return;
  }
  Params p;
  p.x = (const float*)d_in[0];
  p.w = (const float*)d_in[1];
  for (int i = 0; i < NR; ++i) {
    p.out[i] = (float*)d_out_all[i];
    p.ws[i] = (char*)d_ws_all[i];
  }
  p.rank = rank;
  fused<<<GRID, 512, 0, stream>>>(p);
}
```

```cpp
#include <hip/hip_runtime.h>
#include <rccl/rccl.h>
#include <cstdio>
#include <type_traits>

typedef float f32x4 __attribute__((ext_vector_type(4)));
typedef float f32x16 __attribute__((ext_vector_type(16)));
typedef _Float16 f16x8 __attribute__((ext_vector_type(8)));
typedef _Float16 f16x2 __attribute__((ext_vector_type(2)));
typedef float f32x2 __attribute__((ext_vector_type(2)));
typedef unsigned int u32x4 __attribute__((ext_vector_type(4)));
typedef unsigned int u32x2 __attribute__((ext_vector_type(2)));

constexpr int NR = 8;
constexpr int ML = 512;
constexpr int KK = 4096, NN = 4096;
constexpr int NS = 512;
constexpr int ROUNDS = 8;
constexpr int GRID = 256;
constexpr unsigned long long DEADLINE = 20000000ull;
constexpr unsigned OOB = 0xFFFF0000u;

constexpr size_t WS_DONE = 0;
constexpr size_t WS_TMO = 1024;
constexpr size_t WS_CNT = 4096;
constexpr size_t WS_SLAB = 1u << 20;
constexpr size_t SLAB_BYTES = (size_t)128 * 4 * 4 * 4 * 3 * 1024;
constexpr size_t WS_NEED = WS_SLAB + SLAB_BYTES;

constexpr int RING_BYTES = 65536;
constexpr int LDS_DONE = 2 * RING_BYTES;
constexpr int LDS_BYTES = LDS_DONE + 16;

struct Params {
  const float* x;
  const float* w;
  char* myws;
  float* out[NR];
  char* ws[NR];
  int rank;
};

#define LD_NB 2
#define B_LOOK 8
#define ABL_PST false
#define ABL_GLD false
#define PST_AUX 16
#define A_POS 40
#define F_POS 7
#define ISSUE_MODE 0
#define NA_AGPR 64
#define OUT_AUX 17
#define LOCAL_LAST 0
#define KPROG 0
#define W_AUX 0
constexpr int AUX_SC1 = 16;
constexpr int AUX_SYS = 17;

__device__ __forceinline__ float silu(float y) {
  float c = fminf(fmaxf(y, -60.f), 60.f);
  return y * __builtin_amdgcn_rcpf(1.f + __expf(-c));
}

__device__ __forceinline__ unsigned pack2(float a, float b) {
  f32x2 v = {a, b};
  f16x2 h = __builtin_convertvector(v, f16x2);
  return __builtin_bit_cast(unsigned, h);
}

__device__ __forceinline__ void strip_map(int j, int u, int rank, int& dest, int& dcol) {
  const int v = (u + 2 * j) & 15;
  if (v < 14) {
    const int h = v >= 7;
    dest = (rank + 1 + v - 7 * h) & 7;
    dcol = (2 * j + h) * 32;
  } else {
    dest = rank;
    dcol = (2 * j + v - 14) * 32;
  }
}

__device__ __forceinline__ int bslot(int c, int i) {
  return ((c + 2 * i) & 7) + 8 * (i & 1) + 16 * (__builtin_popcount(c) & 1);
}

#define RSTAMP(i) do {} while (0)
#define TICK() 0u
#define ACC(i, v) do {} while (0)
#define STAMP(i) do {} while (0)
#define STAMP_FLUSH() do {} while (0)
#define WT0() do {} while (0)
#define WT1(i) do {} while (0)
#define WCNT(i) do {} while (0)

#define SB() __builtin_amdgcn_sched_barrier(0)

__global__ void __launch_bounds__(256, 1) fused(Params p) {
  __shared__ __attribute__((aligned(16))) char smem[LDS_BYTES];
  const int tid = threadIdx.x, lane = tid & 63;
  const int w = __builtin_amdgcn_readfirstlane(tid >> 6);
  char* const myws = p.myws;
  const int bid = blockIdx.x;
  const int g8 = bid & 7, idx = bid >> 3;
  const int nh = g8 & 1, kq = g8 >> 1;
  const int mi = idx & 3, ng = idx >> 2;
#define ROT_MI 5
  const int rot = (ROT_MI * mi + 8 * (ng >> 2)) & 15;
  const int qk = (kq - mi) & 3;

  STAMP(0);

  const unsigned done0 = __hip_atomic_load((unsigned*)(myws + WS_DONE + g8 * 128), __ATOMIC_RELAXED, __HIP_MEMORY_SCOPE_AGENT);
  if (tid == 0) *(unsigned*)(smem + LDS_DONE) = 0u;

  const __amdgpu_buffer_rsrc_t rw =
      __builtin_amdgcn_make_buffer_rsrc((void*)p.w, 0, (int)((unsigned)KK * NN * 4u), 0x00020000);
  const unsigned voff = (unsigned)(8 * (lane >> 3)) * (NN * 4u) + (unsigned)(lane & 7) * 16u;
  unsigned rowoff[4];
#pragma unroll
  for (int c = 0; c < 4; ++c) rowoff[c] = (unsigned)(kq * 1024 + ((4 * w + c + rot) & 15) * 64) * (NN * 4u);
  unsigned coltab;
  unsigned outlo, outhi;
  {
    int dest, dcol;
    strip_map(lane & 7, nh * 8 + ng, p.rank, dest, dcol);
    coltab = (unsigned)(dest * NS + dcol) * 4u;
    const unsigned long long a = (unsigned long long)p.out[dest & 7] + ((size_t)(p.rank * ML + mi * 128) * NS + dcol) * 4u;
    outlo = (unsigned)a;
    outhi = (unsigned)(a >> 32);
  }
  unsigned wroff[4];
#pragma unroll
  for (int i = 0; i < 4; ++i)
    wroff[i] = (unsigned)(w * 16384 + (lane >> 4) * 1024 + (((lane >> 3) & 1) * 32 + bslot(lane & 7, i)) * 16);
  const unsigned rdoff = (unsigned)(((lane >> 5) * 32 + bslot((lane & 31) >> 2, lane & 3)) * 16);
  f32x4 buf[LD_NB][8];

  auto w_issue = [&](int j, int c) {
    const unsigned base = rowoff[c] + __builtin_amdgcn_readlane(coltab, j);
#pragma unroll
    for (int jj = 0; jj < 8; ++jj)
      buf[c % LD_NB][jj] = __builtin_bit_cast(f32x4, __builtin_amdgcn_raw_buffer_load_b128(rw, voff, base + (unsigned)jj * (NN * 4u), W_AUX));
  };
  auto w_issue1 = [&](unsigned base, int c, int jj) {
    buf[c % LD_NB][jj] = __builtin_bit_cast(f32x4, __builtin_amdgcn_raw_buffer_load_b128(rw, voff, base + (unsigned)jj * (NN * 4u), W_AUX));
  };
  u32x4 pk[4];
  auto w_cvt = [&](int c) {
#pragma unroll
    for (int i = 0; i < 4; ++i) {
#pragma unroll
      for (int h = 0; h < 4; ++h) pk[i][h] = pack2(buf[c % LD_NB][2 * h][i], buf[c % LD_NB][2 * h + 1][i]);
      asm volatile("" : "+v"(pk[i]));
    }
  };
  auto w_write = [&](int j, int c) {
    char* ring = smem + (j & 1) * RING_BYTES + c * 4096;
#pragma unroll
    for (int i = 0; i < 4; ++i) *(u32x4*)(ring + wroff[i]) = pk[i];
  };

  unsigned call = 0, tag = 0;

  const __amdgpu_buffer_rsrc_t rs =
      __builtin_amdgcn_make_buffer_rsrc(myws + WS_SLAB, 0, (int)SLAB_BYTES, 0x00020000);
  unsigned* const tmo = (unsigned*)(myws + WS_TMO);
  f32x4 pend0 = {0.f, 0.f, 0.f, 0.f}, pend1 = pend0;
  u32x4 pr[3];
  unsigned tm = 0;
  int lo = 0;
  bool infl = false;

  auto tile_off = [&](int j, int q) {
    return (unsigned)((((j * 16 + nh * 8 + ng) * 4 + mi) * 4 + w) * 4 + q) * 3u * 1024u;
  };
  auto gather1 = [&](unsigned o, int i) { pr[i] = __builtin_amdgcn_raw_buffer_load_b128(rs, o, (unsigned)i * 1024u, AUX_SC1); };
  const __amdgpu_buffer_rsrc_t rc = __builtin_amdgcn_make_buffer_rsrc(myws, 0, 4096, 0x00020000);
  auto gather_tm = [&]() { tm = __builtin_amdgcn_raw_buffer_load_b32(rc, (unsigned)WS_TMO, 0, AUX_SC1); };
  auto gather_off = [&]() { return tile_off(lo, qk) + (unsigned)lane * 16u; };
  auto tags_ok = [&]() -> bool {
    asm volatile("" : "+v"(tm));
    unsigned all1 = ~0u, any1 = 0u;
#pragma unroll
    for (int i = 0; i < 3; ++i)
#pragma unroll
      for (int e = 0; e < 4; ++e) { all1 &= pr[i][e]; any1 |= pr[i][e]; }
    const bool ok = tag ? (all1 & 1u) : !(any1 & 1u);
    return __all(ok || tm == call);
  };
  auto gather_finish = [&](f32x4& y) {
    if (!tags_ok()) {
      const unsigned long long t0 = __builtin_amdgcn_s_memrealtime();
#pragma unroll 1
      do {
        WCNT(26);
        if (__builtin_amdgcn_s_memrealtime() - t0 > DEADLINE) {
          __hip_atomic_store(tmo, call, __ATOMIC_RELAXED, __HIP_MEMORY_SCOPE_AGENT);
          tm = call;
          break;
        }
        __builtin_amdgcn_s_sleep(1);
        const unsigned o = gather_off();
#pragma unroll
        for (int i = 0; i < 3; ++i) gather1(o, i);
        gather_tm();
      } while (!tags_ok());
    }
    const float poison = __any(tm == call) ? __builtin_nanf("") : 0.f;
    const f32x4 own = (lo & 1) ? pend1 : pend0;
    const f32x4 p0 = __builtin_bit_cast(f32x4, pr[0]), p1 = __builtin_bit_cast(f32x4, pr[1]), p2 = __builtin_bit_cast(f32x4, pr[2]);
#pragma unroll
    for (int e = 0; e < 4; ++e) y[e] = silu(own[e] + p0[e] + p1[e] + p2[e]) + poison;
  };
  const unsigned ovoff = (unsigned)((w * 32 + 8 * qk + 4 * (lane >> 5)) * NS + (lane & 31)) * 4u;
  auto out_rsrc = [&](int j) {
    const unsigned hi = (unsigned)__builtin_amdgcn_readlane(outhi, j), lo32 = (unsigned)__builtin_amdgcn_readlane(outlo, j);
    const unsigned long long a = ((unsigned long long)hi << 32) | (unsigned long long)lo32;
    return __builtin_amdgcn_make_buffer_rsrc((void*)a, 0, 128 * NS * 4, 0x00020000);
  };
  auto out1 = [&](const f32x4& y, __amdgpu_buffer_rsrc_t r, unsigned o, int e) {
    const u32x4 u = __builtin_bit_cast(u32x4, y);
    __builtin_amdgcn_raw_buffer_store_b32(u[e], r, o, (unsigned)e * (NS * 4u), OUT_AUX);
  };
  auto finish_now = [&]() {
    f32x4 y;
    const unsigned o = gather_off();
#pragma unroll
    for (int i = 0; i < 3; ++i) gather1(o, i);
    gather_tm();
    gather_finish(y);
    const __amdgpu_buffer_rsrc_t r = out_rsrc(lo);
#pragma unroll
    for (int e = 0; e < 4; ++e) out1(y, r, ovoff, e);
    STAMP(12 + lo);
    ++lo;
  };

  u32x4 pst[4];
  auto part_prepare = [&](int j, const f32x16& acc) {
#pragma unroll
    for (int q = 0; q < 4; ++q) {
      const f32x4 a = {acc[4 * q], acc[4 * q + 1], acc[4 * q + 2], acc[4 * q + 3]};
      if (q == qk) { if (j & 1) pend1 = a; else pend0 = a; }
      pst[q] = __builtin_bit_cast(u32x4, a);
#pragma unroll
      for (int i = 0; i < 4; ++i) pst[q][i] = (pst[q][i] & ~1u) | tag;
    }
  };
  auto part_store = [&](int j, int q, bool valid) {
    const unsigned off = (!valid || q == qk || ABL_PST) ? OOB
                         : tile_off(j, q) + (unsigned)((kq - mi - q - 1) & 3) * 1024u + (unsigned)lane * 16u;
    __builtin_amdgcn_raw_buffer_store_b128(pst[q], rs, off, 0, PST_AUX);
  };

  f16x8 A[64];
  {
    const __amdgpu_buffer_rsrc_t rx =
        __builtin_amdgcn_make_buffer_rsrc((void*)p.x, 0, (int)((unsigned)ML * KK * 4u), 0x00020000);
    char* stage = smem + RING_BYTES + w * 16384;
    f32x4 t[2][8];
    auto xi = [&](int q) {
      const int ac = ((q >> 2) * 4 + (lane >> 4) + rot) & 15;
      const unsigned vo = (unsigned)(kq * 1024 + ac * 64 + (lane & 15) * 4) * 4u;
#pragma unroll
      for (int r = 0; r < 8; ++r)
        t[q & 1][r] = __builtin_bit_cast(f32x4, __builtin_amdgcn_raw_buffer_load_b128(
                                                    rx, vo, (unsigned)(mi * 128 + w * 32 + ((q + ng) & 3) * 8 + r) * (KK * 4u), 0));
      SB();
    };
    auto xc = [&](int q) {
      const int ch = lane >> 1;
#pragma unroll
      for (int r = 0; r < 8; ++r) {
        const int row = ((q + ng) & 3) * 8 + r;
        u32x2 v = {pack2(t[q & 1][r][0], t[q & 1][r][1]), pack2(t[q & 1][r][2], t[q & 1][r][3])};
        *(u32x2*)(stage + row * 512 + (((ch & 16) | ((ch ^ row) & 15)) << 4) + (lane & 1) * 8) = v;
      }
      SB();
    };
    auto xr = [&](int b) {
      const int r = lane & 31;
      f16x8 v[16];
#pragma unroll
      for (int e = 0; e < 16; ++e) {
        const int ch = 2 * e + (lane >> 5);
        v[e] = *(const f16x8*)(stage + r * 512 + (((ch & 16) | ((ch ^ r) & 15)) << 4));
      }
      SB();
#pragma unroll
      for (int e = 0; e < 16; ++e) {
        if (b * 16 + e < NA_AGPR) asm volatile("" : "=a"(A[b * 16 + e]) : "0"(v[e]));
        else { asm volatile("" : "+v"(v[e])); A[b * 16 + e] = v[e]; }
      }
      SB();
    };
    xi(0); xi(1);
#pragma unroll
    for (int c = 0; c < LD_NB; ++c) { w_issue(0, c); SB(); }
#pragma unroll
    for (int q = 0; q < 16; ++q) {
      xc(q);
      if (q + 2 < 16) xi(q + 2);
      if ((q & 3) == 3) xr(q >> 2);
      if (q >= 4 && q < 12 && (q & 1) == 0) {
        const int c = (q - 4) >> 1;
        w_cvt(c);
        if (c + LD_NB < 4) w_issue(0, c + LD_NB);
        w_write(0, c);
        SB();
      }
      if (q >= 16 - LD_NB) { w_issue(1, q - (16 - LD_NB)); SB(); }
    }
    {
      unsigned d = done0;
      asm volatile("" : "+v"(d));
      call = __builtin_amdgcn_readfirstlane(d / (GRID / 8) + 1u);
      tag = call & 1u;
    }
  }
  asm volatile("s_waitcnt lgkmcnt(0)" ::: "memory");
  __builtin_amdgcn_s_barrier();
  STAMP(1);

  auto round_body = [&](int j, auto next_tag, auto next2_tag) {
    constexpr bool NEXT = decltype(next_tag)::value, NEXT2 = decltype(next2_tag)::value;
    const char* ring = smem + (j & 1) * RING_BYTES + rdoff;
    char* ringn = smem + ((j + 1) & 1) * RING_BYTES;
    const unsigned col1 = __builtin_amdgcn_readlane(coltab, (j + 1) & 7), col2 = __builtin_amdgcn_readlane(coltab, (j + 2) & 7);
    unsigned go = OOB;
    f32x16 acc;
    f32x4 y = {0.f, 0.f, 0.f, 0.f};
    __amdgpu_buffer_rsrc_t ro = rs;
    unsigned oo = OOB;
    f16x8 B[B_LOOK];
#pragma unroll
    for (int e = 0; e < B_LOOK; ++e) B[e] = *(const f16x8*)(ring + e * 1024);
    SB();
    auto seg = [&](auto lo_tag, auto hi_tag) {
      constexpr int LO = decltype(lo_tag)::value, HI = decltype(hi_tag)::value;
#pragma unroll
      for (int f = LO; f < HI; ++f) {
        const int c = f >> 4, g = f & 15;
        if (f == 0) asm volatile("v_mfma_f32_32x32x16_f16 %0, %1, %2, 0" : "=&v"(acc) : "a"(A[f]), "v"(B[f % B_LOOK]));
        else asm volatile("v_mfma_f32_32x32x16_f16 %0, %1, %2, %0" : "+v"(acc) : "a"(A[f]), "v"(B[f % B_LOOK]));
        if (f + B_LOOK < 64) B[f % B_LOOK] = *(const f16x8*)(ring + (f + B_LOOK) * 1024);
        if (NEXT) {
          if (c > 0 && g < 4) *(u32x4*)(ringn + (c - 1) * 4096 + wroff[g]) = pk[g];
          const int h = g >> 2, i = g & 3;
          pk[i][h] = pack2(buf[c % LD_NB][2 * h][i], buf[c % LD_NB][2 * h + 1][i]);
          asm volatile("" : "+v"(pk[i]));
          if (i == 3) {
            if (c + LD_NB < 4) {
              w_issue1(rowoff[c + LD_NB] + col1, c + LD_NB, 2 * h);
              w_issue1(rowoff[c + LD_NB] + col1, c + LD_NB, 2 * h + 1);
            } else if (NEXT2) {
              w_issue1(rowoff[c + LD_NB - 4] + col2, c + LD_NB - 4, 2 * h);
              w_issue1(rowoff[c + LD_NB - 4] + col2, c + LD_NB - 4, 2 * h + 1);
            }
          }
        }
        if (f == 0) part_store(j - 1, 0, j > KPROG);
        if (f == 1) part_store(j - 1, 1, j > KPROG);
        if (f == 2) part_store(j - 1, 2, j > KPROG);
        if (f == 4) part_store(j - 1, 3, j > KPROG);
        if (f >= A_POS && f < A_POS + 3) gather1(go, f - A_POS);
        if (f == A_POS + 4) gather_tm();
        if (f == F_POS + 1) out1(y, ro, oo, 0);
        if (f == F_POS + 2) out1(y, ro, oo, 1);
        if (f == F_POS + 3) out1(y, ro, oo, 2);
        if (f == F_POS + 5) out1(y, ro, oo, 3);
        SB();
      }
    };
    { WT0(); seg(std::integral_constant<int, 0>{}, std::integral_constant<int, F_POS + 1>{}); WT1(24); }
    {
      WT0();
      ro = out_rsrc(lo & 7);
      oo = infl ? ovoff : OOB;
      if (infl) { gather_finish(y); STAMP(12 + lo); ++lo; }
      infl = lo < j;
      go = (infl && !ABL_GLD) ? gather_off() : OOB;
      SB();
      WT1(25);
    }
    { WT0(); seg(std::integral_constant<int, F_POS + 1>{}, std::integral_constant<int, 64>{}); WT1(24); }
    if (NEXT) {
#pragma unroll
      for (int i = 0; i < 4; ++i) *(u32x4*)(ringn + 3 * 4096 + wroff[i]) = pk[i];
      SB();
    }
    asm volatile("s_nop 11" : "+v"(acc));
    RSTAMP(2 + j);
    {
      WT0();
      part_prepare(j, acc);
      if (!NEXT) {
#pragma unroll
        for (int q = 0; q < 4; ++q) part_store(j, q, true);
      }
      SB();
      WT1(28);
    }
    if (NEXT) {
      WT0();
      asm volatile("s_waitcnt lgkmcnt(0)" ::: "memory");
      __builtin_amdgcn_s_barrier();
      WT1(27);
    }
  };
#pragma unroll 1
  for (int j = KPROG; j < ROUNDS - 2; ++j) round_body(j, std::true_type{}, std::true_type{});
  round_body(ROUNDS - 2, std::true_type{}, std::false_type{});
  round_body(ROUNDS - 1, std::false_type{}, std::false_type{});

  {
    if (infl) {
      f32x4 y;
      gather_finish(y);
      const __amdgpu_buffer_rsrc_t r = out_rsrc(lo);
#pragma unroll
      for (int e = 0; e < 4; ++e) out1(y, r, ovoff, e);
      STAMP(12 + lo);
      ++lo;
    }
    while (lo < ROUNDS - LOCAL_LAST) finish_now();
    asm volatile("s_waitcnt vmcnt(0)" ::: "memory");
    unsigned n = 0;
    if (lane == 0)
      n = __hip_atomic_fetch_add((unsigned*)(smem + LDS_DONE), 1u, __ATOMIC_RELAXED, __HIP_MEMORY_SCOPE_WORKGROUP);
    n = __builtin_amdgcn_readfirstlane(n);
    if (n == 3u && lane < NR) {
      if (lane != p.rank)
        __hip_atomic_fetch_add((unsigned*)(p.ws[lane] + WS_CNT + (p.rank * 8 + g8) * 128), 1u, __ATOMIC_RELAXED,
                               __HIP_MEMORY_SCOPE_SYSTEM);
      else
        __hip_atomic_fetch_add((unsigned*)(myws + WS_DONE + g8 * 128), 1u, __ATOMIC_RELAXED, __HIP_MEMORY_SCOPE_AGENT);
    }
    STAMP(20);
    if (LOCAL_LAST) finish_now();
  }

  if (bid == 0 && w == 0) {
    const int src = lane >> 3;
    bool bad = false;
    if (src != p.rank) {
      unsigned* f = (unsigned*)(myws + WS_CNT + lane * 128);
      const unsigned need = call * (GRID / 8);
      const unsigned long long t0 = __builtin_amdgcn_s_memrealtime();
      while (__hip_atomic_load(f, __ATOMIC_RELAXED, __HIP_MEMORY_SCOPE_SYSTEM) < need) {
        if (__hip_atomic_load(tmo, __ATOMIC_RELAXED, __HIP_MEMORY_SCOPE_AGENT) == call ||
            __builtin_amdgcn_s_memrealtime() - t0 > DEADLINE) {
          __hip_atomic_store(tmo, call, __ATOMIC_RELAXED, __HIP_MEMORY_SCOPE_AGENT);
          bad = true;
          break;
        }
        __builtin_amdgcn_s_sleep(1);
      }
    }
    __builtin_amdgcn_fence(__ATOMIC_ACQUIRE, "");
    if (__any(bad)) {
      asm volatile("s_waitcnt vmcnt(0)" ::: "memory");
      if (lane == 0) p.out[p.rank][(size_t)p.rank * ML * NS] = __builtin_nanf("");
    }
    STAMP(22);
  }
  STAMP_FLUSH();
}

extern "C" void kernel_launch(int rank, int nranks, ncclComm_t comm, hipStream_t stream,
                              void* const* d_in, const int* in_sizes, int n_in,
                              void* d_out, int out_size, const int* in_dtypes, int out_dtype,
                              void* const* d_out_all, void* const* d_ws_all, size_t ws_bytes) {
  if (nranks != NR || n_in < 2 || in_sizes[0] != ML * KK || in_sizes[1] != KK * NN || out_size != NR * ML * NS ||
      ws_bytes < WS_NEED || (unsigned)rank >= (unsigned)NR) {
    fprintf(stderr, "kernel_launch: unexpected problem shape\n");
    return;
  }
  Params p;
  p.x = (const float*)d_in[0];
  p.w = (const float*)d_in[1];
  for (int i = 0; i < NR; ++i) {
    p.out[i] = (float*)d_out_all[i];
    p.ws[i] = (char*)d_ws_all[i];
  }
  p.myws = (char*)d_ws_all[rank];
  p.rank = rank;
  fused<<<GRID, 256, 0, stream>>>(p);
}
```

```cpp
#include <hip/hip_runtime.h>
#include <rccl/rccl.h>
#include <cstdio>
#include <type_traits>

typedef float f32x4 __attribute__((ext_vector_type(4)));
typedef float f32x16 __attribute__((ext_vector_type(16)));
typedef _Float16 f16x8 __attribute__((ext_vector_type(8)));
typedef _Float16 f16x2 __attribute__((ext_vector_type(2)));
typedef float f32x2 __attribute__((ext_vector_type(2)));
typedef unsigned int u32x4 __attribute__((ext_vector_type(4)));
typedef unsigned int u32x2 __attribute__((ext_vector_type(2)));

constexpr int NR = 8;
constexpr int ML = 512;
constexpr int KK = 4096, NN = 4096;
constexpr int NS = 512;
constexpr int ROUNDS = 8;
constexpr int GRID = 256;
constexpr unsigned long long DEADLINE = 20000000ull;
constexpr unsigned OOB = 0xFFFF0000u;

constexpr size_t WS_DONE = 0;
constexpr size_t WS_TMO = 1024;
constexpr size_t WS_CNT = 4096;
constexpr size_t WS_SLAB = 1u << 20;
constexpr size_t SLAB_BYTES = (size_t)16 * 4 * 8 * 4 * 4 * 3 * 1024;
constexpr size_t WS_NEED = WS_SLAB + SLAB_BYTES;

constexpr int RING_BYTES = 65536;
constexpr int LDS_DONE = 2 * RING_BYTES;
constexpr int LDS_BYTES = LDS_DONE + 16;

constexpr int B_LOOK = 8;
constexpr int A_POS = 32;
constexpr int F_POS = 55;

constexpr int AUX_SC1 = 16;
constexpr int AUX_SYS = 17;

struct Params {
  const float* x;
  const float* w;
  char* myws;
  float* out[NR];
  char* ws[NR];
  int rank;
};

__device__ __forceinline__ float silu(float y) {
  const float c = fminf(fmaxf(y, -60.f), 60.f);
  return y * __builtin_amdgcn_rcpf(1.f + __expf(-c));
}

__device__ __forceinline__ unsigned pack2(float a, float b) {
  const f32x2 v = {a, b};
  return __builtin_bit_cast(unsigned, __builtin_convertvector(v, f16x2));
}

__device__ __forceinline__ void strip_map(int j, int u, int rank, int& dest, int& dcol) {
  const int v = (u + 2 * j) & 15;
  if (v < 14) {
    const int h = v >= 7;
    dest = (rank + 1 + v - 7 * h) & 7;
    dcol = (2 * j + h) * 32;
  } else {
    dest = rank;
    dcol = (2 * j + v - 14) * 32;
  }
}

__device__ __forceinline__ int bslot(int c, int i) {
  return ((c + 2 * i) & 7) + 8 * (i & 1) + 16 * (__builtin_popcount(c) & 1);
}

#define STAMP(i) do {} while (0)
#define STAMP_FLUSH() do {} while (0)

#define SB() __builtin_amdgcn_sched_barrier(0)
template <int V> using IC = std::integral_constant<int, V>;

__global__ void __launch_bounds__(256, 1) fused(Params p) {
  __shared__ __attribute__((aligned(16))) char smem[LDS_BYTES];
  const int tid = threadIdx.x, lane = tid & 63;
  const int w = __builtin_amdgcn_readfirstlane(tid >> 6);
  char* const myws = p.myws;
  const int bid = blockIdx.x;
  const int g8 = bid & 7, idx = bid >> 3;
  const int nh = g8 & 1, kq = g8 >> 1;
  const int mi = idx & 3, ng = idx >> 2;
  const int slot = nh * 8 + ng;
  const int rot = (5 * mi + 8 * (ng >> 2)) & 15;
  const int qk = (kq - mi) & 3;

  STAMP(0);
  if (tid == 0) *(unsigned*)(smem + LDS_DONE) = 0u;

  const __amdgpu_buffer_rsrc_t rw =
      __builtin_amdgcn_make_buffer_rsrc((void*)p.w, 0, (int)((unsigned)KK * NN * 4u), 0x00020000);
  const unsigned voff = (unsigned)(8 * (lane >> 3)) * (NN * 4u) + (unsigned)(lane & 7) * 16u;
  unsigned rowoff[4];
#pragma unroll
  for (int c = 0; c < 4; ++c) rowoff[c] = (unsigned)(kq * 1024 + ((4 * w + c + rot) & 15) * 64) * (NN * 4u);
  unsigned coltab;
  unsigned outlo, outhi;
  {
    int dest, dcol;
    strip_map(lane & 7, slot, p.rank, dest, dcol);
    coltab = (unsigned)(dest * NS + dcol) * 4u;
    const unsigned long long a = (unsigned long long)p.out[dest & 7] + ((size_t)(p.rank * ML + mi * 128) * NS + dcol) * 4u;
    outlo = (unsigned)a;
    outhi = (unsigned)(a >> 32);
  }
  unsigned wroff[4];
#pragma unroll
  for (int i = 0; i < 4; ++i)
    wroff[i] = (unsigned)(w * 16384 + (lane >> 4) * 1024 + (((lane >> 3) & 1) * 32 + bslot(lane & 7, i)) * 16);
  const unsigned rdoff = (unsigned)(((lane >> 5) * 32 + bslot((lane & 31) >> 2, lane & 3)) * 16);
  f32x4 buf[2][8];
  u32x4 pk[4];

  auto w_issue1 = [&](unsigned base, int c, int jj) {
    buf[c & 1][jj] = __builtin_bit_cast(f32x4, __builtin_amdgcn_raw_buffer_load_b128(rw, voff, base + (unsigned)jj * (NN * 4u), 0));
  };
  auto w_issue = [&](int j, int c) {
    const unsigned base = rowoff[c] + (unsigned)__builtin_amdgcn_readlane(coltab, j);
#pragma unroll
    for (int jj = 0; jj < 8; ++jj) w_issue1(base, c, jj);
    SB();
  };

  const __amdgpu_buffer_rsrc_t rs =
      __builtin_amdgcn_make_buffer_rsrc(myws + WS_SLAB, 0, (int)SLAB_BYTES, 0x00020000);
  const __amdgpu_buffer_rsrc_t rc = __builtin_amdgcn_make_buffer_rsrc(myws, 0, 4096, 0x00020000);
  unsigned* const tmo = (unsigned*)(myws + WS_TMO);
  unsigned call = 0, tag = 0;
  f32x4 pend0 = {0.f, 0.f, 0.f, 0.f}, pend1 = pend0;
  u32x4 pr[3];
  unsigned tm = 0;
  int lo = 0;

  auto tile_off = [&](int j, int q) { return (unsigned)((((slot * 4 + mi) * 8 + j) * 4 + w) * 4 + q) * 3u * 1024u; };
  auto gather1 = [&](unsigned o, int i) { pr[i] = __builtin_amdgcn_raw_buffer_load_b128(rs, o, (unsigned)i * 1024u, AUX_SC1); };
  auto gather_tm = [&]() { tm = __builtin_amdgcn_raw_buffer_load_b32(rc, (unsigned)WS_TMO, 0, AUX_SC1); };
  auto gather_off = [&]() { return tile_off(lo, qk) + (unsigned)lane * 16u; };
  auto gather_all = [&]() {
    const unsigned o = gather_off();
#pragma unroll
    for (int i = 0; i < 3; ++i) gather1(o, i);
    gather_tm();
  };
  auto tags_ok = [&]() -> bool {
    asm volatile("" : "+v"(tm));
    unsigned all1 = ~0u, any1 = 0u;
#pragma unroll
    for (int i = 0; i < 3; ++i)
#pragma unroll
      for (int e = 0; e < 4; ++e) { all1 &= pr[i][e]; any1 |= pr[i][e]; }
    const bool ok = tag ? (all1 & 1u) : !(any1 & 1u);
    return __all(ok || tm == call);
  };
  auto gather_finish = [&](f32x4& y) {
    if (!tags_ok()) {
      const unsigned long long t0 = __builtin_amdgcn_s_memrealtime();
#pragma unroll 1
      do {
        if (__builtin_amdgcn_s_memrealtime() - t0 > DEADLINE) {
          __hip_atomic_store(tmo, call, __ATOMIC_RELAXED, __HIP_MEMORY_SCOPE_AGENT);
          tm = call;
          break;
        }
        __builtin_amdgcn_s_sleep(1);
        gather_all();
      } while (!tags_ok());
    }
    const float poison = __any(tm == call) ? __builtin_nanf("") : 0.f;
    const f32x4 own = (lo & 1) ? pend1 : pend0;
    const f32x4 p0 = __builtin_bit_cast(f32x4, pr[0]), p1 = __builtin_bit_cast(f32x4, pr[1]), p2 = __builtin_bit_cast(f32x4, pr[2]);
#pragma unroll
    for (int e = 0; e < 4; ++e) y[e] = silu(own[e] + p0[e] + p1[e] + p2[e]) + poison;
  };
  const unsigned ovoff = (unsigned)((w * 32 + 8 * qk + 4 * (lane >> 5)) * NS + (lane & 31)) * 4u;
  auto out_rsrc = [&](int j) {
    const unsigned hi = (unsigned)__builtin_amdgcn_readlane(outhi, j), lo32 = (unsigned)__builtin_amdgcn_readlane(outlo, j);
    const unsigned long long a = ((unsigned long long)hi << 32) | (unsigned long long)lo32;
    return __builtin_amdgcn_make_buffer_rsrc((void*)a, 0, 128 * NS * 4, 0x00020000);
  };
  auto out1 = [&](const f32x4& y, __amdgpu_buffer_rsrc_t r, unsigned o, int e) {
    const u32x4 u = __builtin_bit_cast(u32x4, y);
    __builtin_amdgcn_raw_buffer_store_b32(u[e], r, o, (unsigned)e * (NS * 4u), AUX_SYS);
  };
  auto part_out = [&](int j, const f32x16& acc) {
#pragma unroll
    for (int q = 0; q < 4; ++q) {
      const f32x4 a = {acc[4 * q], acc[4 * q + 1], acc[4 * q + 2], acc[4 * q + 3]};
      if (q == qk) { if (j & 1) pend1 = a; else pend0 = a; }
      u32x4 v = __builtin_bit_cast(u32x4, a);
#pragma unroll
      for (int i = 0; i < 4; ++i) v[i] = (v[i] & ~1u) | tag;
      const unsigned off = q == qk ? OOB : tile_off(j, q) + (unsigned)((kq - mi - q - 1) & 3) * 1024u + (unsigned)lane * 16u;
      __builtin_amdgcn_raw_buffer_store_b128(v, rs, off, 0, AUX_SC1);
    }
    SB();
  };

  f16x8 A[64];
  unsigned touch = 0;
  {
    const __amdgpu_buffer_rsrc_t rx =
        __builtin_amdgcn_make_buffer_rsrc((void*)p.x, 0, (int)((unsigned)ML * KK * 4u), 0x00020000);
    char* stage = smem + RING_BYTES + w * 16384;
    f32x4 t[2][8];
    auto xi = [&](int q) {
      const int ac = ((q >> 2) * 4 + (lane >> 4) + rot) & 15;
      const unsigned vo = (unsigned)(kq * 1024 + ac * 64 + (lane & 15) * 4) * 4u;
#pragma unroll
      for (int r = 0; r < 8; ++r)
        t[q & 1][r] = __builtin_bit_cast(f32x4, __builtin_amdgcn_raw_buffer_load_b128(
                                                    rx, vo, (unsigned)(mi * 128 + w * 32 + ((q + ng) & 3) * 8 + r) * (KK * 4u), 0));
      SB();
    };
    auto xc = [&](int q) {
      const int ch = lane >> 1;
#pragma unroll
      for (int r = 0; r < 8; ++r) {
        const int row = ((q + ng) & 3) * 8 + r;
        const u32x2 v = {pack2(t[q & 1][r][0], t[q & 1][r][1]), pack2(t[q & 1][r][2], t[q & 1][r][3])};
        *(u32x2*)(stage + row * 512 + (((ch & 16) | ((ch ^ row) & 15)) << 4) + (lane & 1) * 8) = v;
      }
      SB();
    };
    auto xr = [&](int b) {
      const int r = lane & 31;
      f16x8 v[16];
#pragma unroll
      for (int e = 0; e < 16; ++e) {
        const int ch = 2 * e + (lane >> 5);
        v[e] = *(const f16x8*)(stage + r * 512 + (((ch & 16) | ((ch ^ r) & 15)) << 4));
      }
      SB();
#pragma unroll
      for (int e = 0; e < 16; ++e) asm volatile("" : "=a"(A[b * 16 + e]) : "0"(v[e]));
      SB();
    };
    unsigned done0 = 0;
    xi(0); xi(1);
    w_issue(0, 0); w_issue(0, 1);
#pragma unroll
    for (int q = 0; q < 16; ++q) {
      xc(q);
      if (q + 2 < 16) xi(q + 2);
      if (q == 6) done0 = __builtin_amdgcn_raw_buffer_load_b32(rc, (unsigned)(WS_DONE + g8 * 128), 0, AUX_SC1);
      if ((q & 3) == 3) xr(q >> 2);
      if (q >= 4 && q < 12 && (q & 1) == 0) {
        const int c = (q - 4) >> 1;
#pragma unroll
        for (int i = 0; i < 4; ++i) {
#pragma unroll
          for (int h = 0; h < 4; ++h) pk[i][h] = pack2(buf[c & 1][2 * h][i], buf[c & 1][2 * h + 1][i]);
          asm volatile("" : "+v"(pk[i]));
        }
        if (c + 2 < 4) w_issue(0, c + 2);
#pragma unroll
        for (int i = 0; i < 4; ++i) *(u32x4*)(smem + c * 4096 + wroff[i]) = pk[i];
        SB();
      }
      if (q >= 14) w_issue(1, q - 14);
    }
    if (w == 0) {
      const unsigned long long a = lane < 8 ? (((unsigned long long)outhi << 32) | outlo)
                                            : (unsigned long long)(myws + WS_SLAB + tile_off(0, 0));
      if (lane < 9)
        touch = __hip_atomic_load((const __attribute__((address_space(1))) unsigned*)a, __ATOMIC_RELAXED, __HIP_MEMORY_SCOPE_SYSTEM);
    }
    asm volatile("" : "+v"(done0));
    call = __builtin_amdgcn_readfirstlane(done0 / (GRID / 8) + 1u);
    tag = call & 1u;
  }
  asm volatile("s_waitcnt lgkmcnt(0)" ::: "memory");
  __builtin_amdgcn_s_barrier();
  STAMP(1);

  auto round_body = [&](int j, auto next_tag, auto next2_tag) {
    constexpr bool NEXT = decltype(next_tag)::value, NEXT2 = decltype(next2_tag)::value;
    const char* ring = smem + (j & 1) * RING_BYTES + rdoff;
    char* ringn = smem + ((j + 1) & 1) * RING_BYTES;
    const unsigned col1 = (unsigned)__builtin_amdgcn_readlane(coltab, (j + 1) & 7);
    const unsigned col2 = (unsigned)__builtin_amdgcn_readlane(coltab, (j + 2) & 7);
    const bool have = lo < j;
    const unsigned go = have ? gather_off() : OOB;
    f32x16 acc;
    f32x4 y = {0.f, 0.f, 0.f, 0.f};
    __amdgpu_buffer_rsrc_t ro = rs;
    unsigned oo = OOB;
    f16x8 B[B_LOOK];
#pragma unroll
    for (int e = 0; e < B_LOOK; ++e) B[e] = *(const f16x8*)(ring + e * 1024);
    SB();
    auto seg = [&](auto lo_tag, auto hi_tag) {
      constexpr int LO = decltype(lo_tag)::value, HI = decltype(hi_tag)::value;
#pragma unroll
      for (int f = LO; f < HI; ++f) {
        const int c = f >> 4, g = f & 15;
        if (f == 0) asm volatile("v_mfma_f32_32x32x16_f16 %0, %1, %2, 0" : "=&v"(acc) : "a"(A[f]), "v"(B[f % B_LOOK]));
        else asm volatile("v_mfma_f32_32x32x16_f16 %0, %1, %2, %0" : "+v"(acc) : "a"(A[f]), "v"(B[f % B_LOOK]));
        if (f + B_LOOK < 64) B[f % B_LOOK] = *(const f16x8*)(ring + (f + B_LOOK) * 1024);
        if (NEXT) {
          if (c > 0 && g < 4) *(u32x4*)(ringn + (c - 1) * 4096 + wroff[g]) = pk[g];
          const int h = g >> 2, i = g & 3;
          pk[i][h] = pack2(buf[c & 1][2 * h][i], buf[c & 1][2 * h + 1][i]);
          asm volatile("" : "+v"(pk[i]));
          if (i == 3) {
            if (c < 2) {
              w_issue1(rowoff[c + 2] + col1, c, 2 * h);
              w_issue1(rowoff[c + 2] + col1, c, 2 * h + 1);
            } else if (NEXT2) {
              w_issue1(rowoff[c - 2] + col2, c, 2 * h);
              w_issue1(rowoff[c - 2] + col2, c, 2 * h + 1);
            }
          }
        }
        if (f >= A_POS && f < A_POS + 3) gather1(go, f - A_POS);
        if (f == A_POS + 4) gather_tm();
        if (f == F_POS + 1) out1(y, ro, oo, 0);
        if (f == F_POS + 2) out1(y, ro, oo, 1);
        if (f == F_POS + 3) out1(y, ro, oo, 2);
        if (f == F_POS + 5) out1(y, ro, oo, 3);
        SB();
      }
    };
    seg(IC<0>{}, IC<F_POS + 1>{});
    ro = out_rsrc(lo & 7);
    oo = have ? ovoff : OOB;
    if (have) { gather_finish(y); STAMP(12 + lo); ++lo; }
    SB();
    seg(IC<F_POS + 1>{}, IC<64>{});
    if (NEXT) {
#pragma unroll
      for (int i = 0; i < 4; ++i) *(u32x4*)(ringn + 3 * 4096 + wroff[i]) = pk[i];
      SB();
    }
    asm volatile("s_nop 11" : "+v"(acc));
    part_out(j, acc);
    if (NEXT) {
      asm volatile("s_waitcnt lgkmcnt(0)" ::: "memory");
      __builtin_amdgcn_s_barrier();
    }
  };
#pragma unroll 1
  for (int j = 0; j < ROUNDS - 2; ++j) round_body(j, std::true_type{}, std::true_type{});
  round_body(ROUNDS - 2, std::true_type{}, std::false_type{});
  round_body(ROUNDS - 1, std::false_type{}, std::false_type{});
  STAMP(9);

  {
    f32x4 y;
    __builtin_amdgcn_s_sleep(10);
    gather_all();
    gather_finish(y);
    const __amdgpu_buffer_rsrc_t r = out_rsrc(ROUNDS - 1);
#pragma unroll
    for (int e = 0; e < 4; ++e) out1(y, r, ovoff, e);
    STAMP(12 + ROUNDS - 1);
    asm volatile("s_waitcnt vmcnt(0)" ::: "memory");
    unsigned n = 0;
    if (lane == 0)
      n = __hip_atomic_fetch_add((unsigned*)(smem + LDS_DONE), 1u, __ATOMIC_RELAXED, __HIP_MEMORY_SCOPE_WORKGROUP);
    n = __builtin_amdgcn_readfirstlane(n);
    if (n == 3u && lane < NR) {
      if (lane != p.rank)
        __hip_atomic_fetch_add((unsigned*)(p.ws[lane] + WS_CNT + (p.rank * 8 + g8) * 128), 1u, __ATOMIC_RELAXED,
                               __HIP_MEMORY_SCOPE_SYSTEM);
      else
        __hip_atomic_fetch_add((unsigned*)(myws + WS_DONE + g8 * 128), 1u, __ATOMIC_RELAXED, __HIP_MEMORY_SCOPE_AGENT);
    }
    STAMP(20);
  }
  asm volatile("" :: "v"(touch));

  if (bid == 0 && w == 0) {
    const int src = lane >> 3;
    bool bad = false;
    unsigned* f = (unsigned*)(myws + WS_CNT + lane * 128);
    const unsigned need = call * (GRID / 8);
    const unsigned long long t0 = __builtin_amdgcn_s_memrealtime();
#pragma unroll 1
    for (unsigned spin = 1;; ++spin) {
      const unsigned v = __hip_atomic_load(f, __ATOMIC_RELAXED, __HIP_MEMORY_SCOPE_SYSTEM);
      if (__all(src == p.rank || v >= need)) break;
      if ((spin & 7u) == 0u) {
        if (__hip_atomic_load(tmo, __ATOMIC_RELAXED, __HIP_MEMORY_SCOPE_AGENT) == call ||
            __builtin_amdgcn_s_memrealtime() - t0 > DEADLINE) {
          __hip_atomic_store(tmo, call, __ATOMIC_RELAXED, __HIP_MEMORY_SCOPE_AGENT);
          bad = true;
          break;
        }
      }
      __builtin_amdgcn_s_sleep(1);
    }
    __builtin_amdgcn_fence(__ATOMIC_ACQUIRE, "");
    if (bad) {
      asm volatile("s_waitcnt vmcnt(0)" ::: "memory");
      if (lane == 0) p.out[p.rank][(size_t)p.rank * ML * NS] = __builtin_nanf("");
    }
    STAMP(22);
  }
  STAMP_FLUSH();
}

extern "C" void kernel_launch(int rank, int nranks, ncclComm_t comm, hipStream_t stream,
                              void* const* d_in, const int* in_sizes, int n_in,
                              void* d_out, int out_size, const int* in_dtypes, int out_dtype,
                              void* const* d_out_all, void* const* d_ws_all, size_t ws_bytes) {
  if (nranks != NR || n_in < 2 || in_sizes[0] != ML * KK || in_sizes[1] != KK * NN || out_size != NR * ML * NS ||
      ws_bytes < WS_NEED || (unsigned)rank >= (unsigned)NR) {
    fprintf(stderr, "kernel_launch: unexpected problem shape\n");
    return;
  }
  Params p;
  p.x = (const float*)d_in[0];
  p.w = (const float*)d_in[1];
  for (int i = 0; i < NR; ++i) {
    p.out[i] = (float*)d_out_all[i];
    p.ws[i] = (char*)d_ws_all[i];
  }
  p.myws = (char*)d_ws_all[rank];
  p.rank = rank;
  fused<<<GRID, 256, 0, stream>>>(p);
}
```

```cpp
#include <hip/hip_runtime.h>
#include <rccl/rccl.h>
#include <cstddef>
#include <cstdio>
#include <type_traits>

typedef float f32x4 __attribute__((ext_vector_type(4)));
typedef float f32x16 __attribute__((ext_vector_type(16)));
typedef _Float16 f16x8 __attribute__((ext_vector_type(8)));
typedef _Float16 f16x2 __attribute__((ext_vector_type(2)));
typedef float f32x2 __attribute__((ext_vector_type(2)));
typedef unsigned int u32x4 __attribute__((ext_vector_type(4)));
typedef unsigned int u32x2 __attribute__((ext_vector_type(2)));
typedef unsigned int u32x8 __attribute__((ext_vector_type(8)));

constexpr int NR = 8;
constexpr int ML = 512;
constexpr int KK = 4096, NN = 4096;
constexpr int NS = 512;
constexpr int ROUNDS = 8;
constexpr int GRID = 256;
constexpr unsigned long long DEADLINE = 20000000ull;
constexpr unsigned OOB = 0xFFFF0000u;

constexpr size_t WS_DONE = 0;
constexpr size_t WS_TMO = 1024;
constexpr size_t WS_CNT = 4096;
constexpr size_t WS_SLAB = 1u << 20;
constexpr size_t SLAB_BYTES = (size_t)16 * 4 * 8 * 4 * 4 * 3 * 1024;
constexpr size_t WS_NEED = WS_SLAB + SLAB_BYTES;

constexpr int RING_BYTES = 65536;
constexpr int LDS_DONE = 2 * RING_BYTES;
constexpr int LDS_BYTES = LDS_DONE + 16;

constexpr int B_LOOK = 8;
constexpr int A_POS = 32;
constexpr int F_POS = 55;

constexpr int AUX_SC1 = 16;
constexpr int AUX_SYS = 17;

struct Params {
  const float* x;
  const float* w;
  char* myws;
  int rank, pad;
  float* out[NR];
  char* ws[NR];
};
static_assert(offsetof(Params, x) == 0 && offsetof(Params, w) == 8 && offsetof(Params, myws) == 16 && offsetof(Params, rank) == 24,
              "the kernel reads the first 32 bytes of its arguments in one piece");

__device__ __forceinline__ float silu(float y) {
  const float c = fminf(fmaxf(y, -60.f), 60.f);
  return y * __builtin_amdgcn_rcpf(1.f + __expf(-c));
}

__device__ __forceinline__ unsigned pack2(float a, float b) {
  const f32x2 v = {a, b};
  return __builtin_bit_cast(unsigned, __builtin_convertvector(v, f16x2));
}

__device__ __forceinline__ void strip_map(int j, int u, int rank, int& dest, int& dcol) {
  int di, sd;
  if (j < 7) {
    if (u == j) { di = 7; sd = j; }
    else {
      const int t = u - (u > j);
      if (t < 14) { const int h = t >= 7; di = t - 7 * h; sd = 2 * j + h; }
      else { di = j; sd = 14; }
    }
  } else {
    if (u < 7) { di = u; sd = 15; }
    else { di = 7; sd = u; }
  }
  dest = (rank + 1 + di) & 7;
  dcol = sd * 32;
}

__device__ __forceinline__ int bslot(int c, int i) {
  return ((c + 2 * i) & 7) + 8 * (i & 1) + 16 * (__builtin_popcount(c) & 1);
}

#define SB() __builtin_amdgcn_sched_barrier(0)
template <int V> using IC = std::integral_constant<int, V>;

__global__ void __launch_bounds__(256, 1) fused(Params p) {
  __shared__ __attribute__((aligned(16))) char smem[LDS_BYTES];
  const int tid = threadIdx.x, lane = tid & 63;
  const int w = __builtin_amdgcn_readfirstlane(tid >> 6);
  const u32x8 hdr = *(const __attribute__((address_space(4))) u32x8*)__builtin_amdgcn_kernarg_segment_ptr();
  const float* const px = (const float*)(((unsigned long long)hdr[1] << 32) | hdr[0]);
  const float* const pw = (const float*)(((unsigned long long)hdr[3] << 32) | hdr[2]);
  char* const myws = (char*)(__attribute__((address_space(1))) char*)(((unsigned long long)hdr[5] << 32) | hdr[4]);
  const int rank = (int)hdr[6];
  const int bid = blockIdx.x;
  const int g8 = bid & 7, idx = bid >> 3;
  const int nh = g8 & 1, kq = g8 >> 1;
  const int mi = idx & 3, ng = idx >> 2;
  const int slot = nh * 8 + ng;
  const int rot = (5 * mi + 8 * (ng >> 2)) & 15;
  const int qk = (kq - mi) & 3;

  const __amdgpu_buffer_rsrc_t rx =
      __builtin_amdgcn_make_buffer_rsrc((void*)px, 0, (int)((unsigned)ML * KK * 4u), 0x00020000);
  f32x4 t[2][8];
  auto xi = [&](int q) {
    const int ac = ((q >> 2) * 4 + (lane >> 4) + rot) & 15;
    const unsigned vo = (unsigned)(kq * 1024 + ac * 64 + (lane & 15) * 4) * 4u;
#pragma unroll
    for (int r = 0; r < 8; ++r)
      t[q & 1][r] = __builtin_bit_cast(f32x4, __builtin_amdgcn_raw_buffer_load_b128(
                                                  rx, vo, (unsigned)(mi * 128 + w * 32 + ((q + ng) & 3) * 8 + r) * (KK * 4u), 0));
    SB();
  };
  xi(0); xi(1);
  if (tid == 0) *(unsigned*)(smem + LDS_DONE) = 0u;

  const __amdgpu_buffer_rsrc_t rw =
      __builtin_amdgcn_make_buffer_rsrc((void*)pw, 0, (int)((unsigned)KK * NN * 4u), 0x00020000);
  const unsigned voff = (unsigned)(8 * (lane >> 3)) * (NN * 4u) + (unsigned)(lane & 7) * 16u;
  unsigned rowoff[4];
#pragma unroll
  for (int c = 0; c < 4; ++c) rowoff[c] = (unsigned)(kq * 1024 + ((4 * w + c + rot) & 15) * 64) * (NN * 4u);
  int tdest, tdcol;
  strip_map(lane & 7, slot, rank, tdest, tdcol);
  const unsigned coltab = (unsigned)(tdest * NS + tdcol) * 4u;
  unsigned wroff[4];
#pragma unroll
  for (int i = 0; i < 4; ++i)
    wroff[i] = (unsigned)(w * 16384 + (lane >> 4) * 1024 + (((lane >> 3) & 1) * 32 + bslot(lane & 7, i)) * 16);
  const unsigned rdoff = (unsigned)(((lane >> 5) * 32 + bslot((lane & 31) >> 2, lane & 3)) * 16);
  f32x4 buf[2][8];
  u32x4 pk[4];

  auto w_issue1 = [&](unsigned base, int c, int jj) {
    buf[c & 1][jj] = __builtin_bit_cast(f32x4, __builtin_amdgcn_raw_buffer_load_b128(rw, voff, base + (unsigned)jj * (NN * 4u), 0));
  };
  auto w_issue = [&](int j, int c) {
    const unsigned base = rowoff[c] + (unsigned)__builtin_amdgcn_readlane(coltab, j);
#pragma unroll
    for (int jj = 0; jj < 8; ++jj) w_issue1(base, c, jj);
    SB();
  };

  w_issue(0, 0); w_issue(0, 1);
  unsigned long long optr = (unsigned long long)p.out[tdest & 7];
  unsigned outlo = 0, outhi = 0;
  SB();

  const __amdgpu_buffer_rsrc_t rs =
      __builtin_amdgcn_make_buffer_rsrc(myws + WS_SLAB, 0, (int)SLAB_BYTES, 0x00020000);
  const __amdgpu_buffer_rsrc_t rc = __builtin_amdgcn_make_buffer_rsrc(myws, 0, 4096, 0x00020000);
  unsigned* const tmo = (unsigned*)(myws + WS_TMO);
  unsigned call = 0, tag = 0;
  f32x4 pend0 = {0.f, 0.f, 0.f, 0.f}, pend1 = pend0;
  u32x4 pr[3];
  unsigned tm = 0;
  int lo = 0;

  auto tile_off = [&](int j, int q) { return (unsigned)((((slot * 4 + mi) * 8 + j) * 4 + w) * 4 + q) * 3u * 1024u; };
  auto gather1 = [&](unsigned o, int i) { pr[i] = __builtin_amdgcn_raw_buffer_load_b128(rs, o, (unsigned)i * 1024u, AUX_SC1); };
  auto gather_tm = [&]() { tm = __builtin_amdgcn_raw_buffer_load_b32(rc, (unsigned)WS_TMO, 0, AUX_SC1); };
  auto gather_off = [&]() { return tile_off(lo, qk) + (unsigned)lane * 16u; };
  auto gather_all = [&]() {
    const unsigned o = gather_off();
#pragma unroll
    for (int i = 0; i < 3; ++i) gather1(o, i);
    gather_tm();
  };
  auto tags_ok = [&]() -> bool {
    asm volatile("" : "+v"(tm));
    unsigned all1 = ~0u, any1 = 0u;
#pragma unroll
    for (int i = 0; i < 3; ++i)
#pragma unroll
      for (int e = 0; e < 4; ++e) { all1 &= pr[i][e]; any1 |= pr[i][e]; }
    const bool ok = tag ? (all1 & 1u) : !(any1 & 1u);
    return __all(ok || tm == call);
  };
  auto gather_finish = [&](f32x4& y) {
    if (!tags_ok()) {
      const unsigned long long t0 = __builtin_amdgcn_s_memrealtime();
#pragma unroll 1
      do {
        if (__builtin_amdgcn_s_memrealtime() - t0 > DEADLINE) {
          __hip_atomic_store(tmo, call, __ATOMIC_RELAXED, __HIP_MEMORY_SCOPE_AGENT);
          tm = call;
          break;
        }
        __builtin_amdgcn_s_sleep(1);
        gather_all();
      } while (!tags_ok());
    }
    const float poison = __any(tm == call) ? __builtin_nanf("") : 0.f;
    const f32x4 own = (lo & 1) ? pend1 : pend0;
    const f32x4 p0 = __builtin_bit_cast(f32x4, pr[0]), p1 = __builtin_bit_cast(f32x4, pr[1]), p2 = __builtin_bit_cast(f32x4, pr[2]);
#pragma unroll
    for (int e = 0; e < 4; ++e) y[e] = silu(own[e] + p0[e] + p1[e] + p2[e]) + poison;
  };
  const unsigned ovoff = (unsigned)((w * 32 + 8 * qk + 4 * (lane >> 5)) * NS + (lane & 31)) * 4u;
  auto out_rsrc = [&](int j) {
    const unsigned hi = (unsigned)__builtin_amdgcn_readlane(outhi, j), lo32 = (unsigned)__builtin_amdgcn_readlane(outlo, j);
    const unsigned long long a = ((unsigned long long)hi << 32) | (unsigned long long)lo32;
    return __builtin_amdgcn_make_buffer_rsrc((void*)a, 0, 128 * NS * 4, 0x00020000);
  };
  auto out1 = [&](const f32x4& y, __amdgpu_buffer_rsrc_t r, unsigned o, int e) {
    const u32x4 u = __builtin_bit_cast(u32x4, y);
    __builtin_amdgcn_raw_buffer_store_b32(u[e], r, o, (unsigned)e * (NS * 4u), AUX_SYS);
  };
  auto part_out = [&](int j, const f32x16& acc) {
#pragma unroll
    for (int q = 0; q < 4; ++q) {
      const f32x4 a = {acc[4 * q], acc[4 * q + 1], acc[4 * q + 2], acc[4 * q + 3]};
      if (q == qk) { if (j & 1) pend1 = a; else pend0 = a; }
      u32x4 v = __builtin_bit_cast(u32x4, a);
#pragma unroll
      for (int i = 0; i < 4; ++i) v[i] = (v[i] & ~1u) | tag;
      const unsigned off = q == qk ? OOB : tile_off(j, q) + (unsigned)((kq - mi - q - 1) & 3) * 1024u + (unsigned)lane * 16u;
      __builtin_amdgcn_raw_buffer_store_b128(v, rs, off, 0, AUX_SC1);
    }
    SB();
  };

  f16x8 A[64];
  unsigned touch = 0;
  {
    char* stage = smem + RING_BYTES + w * 16384;
    auto xc = [&](int q) {
      const int ch = lane >> 1;
#pragma unroll
      for (int r = 0; r < 8; ++r) {
        const int row = ((q + ng) & 3) * 8 + r;
        const u32x2 v = {pack2(t[q & 1][r][0], t[q & 1][r][1]), pack2(t[q & 1][r][2], t[q & 1][r][3])};
        *(u32x2*)(stage + row * 512 + (((ch & 16) | ((ch ^ row) & 15)) << 4) + (lane & 1) * 8) = v;
      }
      SB();
    };
    auto xr = [&](int b) {
      const int r = lane & 31;
      f16x8 v[16];
#pragma unroll
      for (int e = 0; e < 16; ++e) {
        const int ch = 2 * e + (lane >> 5);
        v[e] = *(const f16x8*)(stage + r * 512 + (((ch & 16) | ((ch ^ r) & 15)) << 4));
      }
      SB();
#pragma unroll
      for (int e = 0; e < 16; ++e) asm volatile("" : "=a"(A[b * 16 + e]) : "0"(v[e]));
      SB();
    };
    unsigned done0 = 0;
#pragma unroll
    for (int q = 0; q < 16; ++q) {
      xc(q);
      if (q + 2 < 16) xi(q + 2);
      if (q == 8) {
        asm volatile("" : "+v"(optr));
        const unsigned long long a = optr + ((size_t)(rank * ML + mi * 128) * NS + tdcol) * 4u;
        outlo = (unsigned)a;
        outhi = (unsigned)(a >> 32);
      }
      if (q == 6) done0 = __builtin_amdgcn_raw_buffer_load_b32(rc, (unsigned)(WS_DONE + g8 * 128), 0, AUX_SC1);
      if ((q & 3) == 3) xr(q >> 2);
      if (q >= 4 && q < 12 && (q & 1) == 0) {
        const int c = (q - 4) >> 1;
#pragma unroll
        for (int i = 0; i < 4; ++i) {
#pragma unroll
          for (int h = 0; h < 4; ++h) pk[i][h] = pack2(buf[c & 1][2 * h][i], buf[c & 1][2 * h + 1][i]);
          asm volatile("" : "+v"(pk[i]));
        }
        if (c + 2 < 4) w_issue(0, c + 2);
#pragma unroll
        for (int i = 0; i < 4; ++i) *(u32x4*)(smem + c * 4096 + wroff[i]) = pk[i];
        SB();
      }
      if (q >= 14) w_issue(1, q - 14);
    }
    if (w == 0) {
      const unsigned long long a = lane < 8 ? (((unsigned long long)outhi << 32) | outlo)
                                            : (unsigned long long)(myws + WS_SLAB + tile_off(0, 0));
      if (lane < 9)
        touch = __hip_atomic_load((const __attribute__((address_space(1))) unsigned*)a, __ATOMIC_RELAXED, __HIP_MEMORY_SCOPE_SYSTEM);
    }
    asm volatile("" : "+v"(done0));
    call = __builtin_amdgcn_readfirstlane(done0 / (GRID / 8) + 1u);
    tag = call & 1u;
  }
  asm volatile("s_waitcnt lgkmcnt(0)" ::: "memory");
  __builtin_amdgcn_s_barrier();

  auto round_body = [&](int j, auto next_tag, auto next2_tag) {
    constexpr bool NEXT = decltype(next_tag)::value, NEXT2 = decltype(next2_tag)::value;
    const char* ring = smem + (j & 1) * RING_BYTES + rdoff;
    char* ringn = smem + ((j + 1) & 1) * RING_BYTES;
    const unsigned col1 = (unsigned)__builtin_amdgcn_readlane(coltab, (j + 1) & 7);
    const unsigned col2 = (unsigned)__builtin_amdgcn_readlane(coltab, (j + 2) & 7);
    const bool have = lo < j;
    const unsigned go = have ? gather_off() : OOB;
    f32x16 acc;
    f32x4 y = {0.f, 0.f, 0.f, 0.f};
    __amdgpu_buffer_rsrc_t ro = rs;
    unsigned oo = OOB;
    f16x8 B[B_LOOK];
#pragma unroll
    for (int e = 0; e < B_LOOK; ++e) B[e] = *(const f16x8*)(ring + e * 1024);
    SB();
    auto seg = [&](auto lo_tag, auto hi_tag) {
      constexpr int LO = decltype(lo_tag)::value, HI = decltype(hi_tag)::value;
#pragma unroll
      for (int f = LO; f < HI; ++f) {
        const int c = f >> 4, g = f & 15;
        if (f == 0) asm volatile("v_mfma_f32_32x32x16_f16 %0, %1, %2, 0" : "=&v"(acc) : "a"(A[f]), "v"(B[f % B_LOOK]));
        else asm volatile("v_mfma_f32_32x32x16_f16 %0, %1, %2, %0" : "+v"(acc) : "a"(A[f]), "v"(B[f % B_LOOK]));
        if (f + B_LOOK < 64) B[f % B_LOOK] = *(const f16x8*)(ring + (f + B_LOOK) * 1024);
        if (NEXT) {
          if (c > 0 && g < 4) *(u32x4*)(ringn + (c - 1) * 4096 + wroff[g]) = pk[g];
          const int h = g >> 2, i = g & 3;
          pk[i][h] = pack2(buf[c & 1][2 * h][i], buf[c & 1][2 * h + 1][i]);
          asm volatile("" : "+v"(pk[i]));
          if (i == 3) {
            if (c < 2) {
              w_issue1(rowoff[c + 2] + col1, c, 2 * h);
              w_issue1(rowoff[c + 2] + col1, c, 2 * h + 1);
            } else if (NEXT2) {
              w_issue1(rowoff[c - 2] + col2, c, 2 * h);
              w_issue1(rowoff[c - 2] + col2, c, 2 * h + 1);
            }
          }
        }
        if (f >= A_POS && f < A_POS + 3) gather1(go, f - A_POS);
        if (f == A_POS + 4) gather_tm();
        if (f == F_POS + 1) out1(y, ro, oo, 0);
        if (f == F_POS + 2) out1(y, ro, oo, 1);
        if (f == F_POS + 3) out1(y, ro, oo, 2);
        if (f == F_POS + 5) out1(y, ro, oo, 3);
        SB();
      }
    };
    seg(IC<0>{}, IC<F_POS + 1>{});
    ro = out_rsrc(lo & 7);
    oo = have ? ovoff : OOB;
    if (have) { gather_finish(y); ++lo; }
    SB();
    seg(IC<F_POS + 1>{}, IC<64>{});
    if (NEXT) {
#pragma unroll
      for (int i = 0; i < 4; ++i) *(u32x4*)(ringn + 3 * 4096 + wroff[i]) = pk[i];
      SB();
    }
    asm volatile("s_nop 11" : "+v"(acc));
    part_out(j, acc);
    if (NEXT) {
      asm volatile("s_waitcnt lgkmcnt(0)" ::: "memory");
      __builtin_amdgcn_s_barrier();
    }
  };
#pragma unroll 1
  for (int j = 0; j < ROUNDS - 2; ++j) round_body(j, std::true_type{}, std::true_type{});
  round_body(ROUNDS - 2, std::true_type{}, std::false_type{});
  round_body(ROUNDS - 1, std::false_type{}, std::false_type{});

  {
    f32x4 y;
    __builtin_amdgcn_s_sleep(10);
    gather_all();
    gather_finish(y);
    const __amdgpu_buffer_rsrc_t r = out_rsrc(ROUNDS - 1);
#pragma unroll
    for (int e = 0; e < 4; ++e) out1(y, r, ovoff, e);
    asm volatile("s_waitcnt vmcnt(0)" ::: "memory");
    unsigned n = 0;
    if (lane == 0)
      n = __hip_atomic_fetch_add((unsigned*)(smem + LDS_DONE), 1u, __ATOMIC_RELAXED, __HIP_MEMORY_SCOPE_WORKGROUP);
    n = __builtin_amdgcn_readfirstlane(n);
    if (n == 3u && lane < NR) {
      if (lane != rank)
        __hip_atomic_fetch_add((unsigned*)(p.ws[lane] + WS_CNT + (rank * 8 + g8) * 128), 1u, __ATOMIC_RELAXED,
                               __HIP_MEMORY_SCOPE_SYSTEM);
      else
        __hip_atomic_fetch_add((unsigned*)(myws + WS_DONE + g8 * 128), 1u, __ATOMIC_RELAXED, __HIP_MEMORY_SCOPE_AGENT);
    }
  }
  asm volatile("" :: "v"(touch));

  if (bid == 0 && w == 0) {
    const int src = lane >> 3;
    bool bad = false;
    unsigned* f = (unsigned*)(myws + WS_CNT + lane * 128);
    const unsigned need = call * (GRID / 8);
    const unsigned long long t0 = __builtin_amdgcn_s_memrealtime();
#pragma unroll 1
    for (unsigned spin = 1;; ++spin) {
      const unsigned v = __hip_atomic_load(f, __ATOMIC_RELAXED, __HIP_MEMORY_SCOPE_SYSTEM);
      if (__all(src == rank || v >= need)) break;
      if ((spin & 7u) == 0u) {
        if (__hip_atomic_load(tmo, __ATOMIC_RELAXED, __HIP_MEMORY_SCOPE_AGENT) == call ||
            __builtin_amdgcn_s_memrealtime() - t0 > DEADLINE) {
          __hip_atomic_store(tmo, call, __ATOMIC_RELAXED, __HIP_MEMORY_SCOPE_AGENT);
          bad = true;
          break;
        }
      }
      __builtin_amdgcn_s_sleep(1);
    }
    __builtin_amdgcn_fence(__ATOMIC_ACQUIRE, "");
    if (bad) {
      asm volatile("s_waitcnt vmcnt(0)" ::: "memory");
      if (lane == 0) p.out[rank][(size_t)rank * ML * NS] = __builtin_nanf("");
    }
  }
}

extern "C" void kernel_launch(int rank, int nranks, ncclComm_t comm, hipStream_t stream,
                              void* const* d_in, const int* in_sizes, int n_in,
                              void* d_out, int out_size, const int* in_dtypes, int out_dtype,
                              void* const* d_out_all, void* const* d_ws_all, size_t ws_bytes) {
  if (nranks != NR || n_in < 2 || in_sizes[0] != ML * KK || in_sizes[1] != KK * NN || out_size != NR * ML * NS ||
      ws_bytes < WS_NEED || (unsigned)rank >= (unsigned)NR) {
    fprintf(stderr, "kernel_launch: unexpected problem shape\n");
    return;
  }
  Params p;
  p.x = (const float*)d_in[0];
  p.w = (const float*)d_in[1];
  for (int i = 0; i < NR; ++i) {
    p.out[i] = (float*)d_out_all[i];
    p.ws[i] = (char*)d_ws_all[i];
  }
  p.myws = (char*)d_ws_all[rank];
  p.rank = rank;
  fused<<<GRID, 256, 0, stream>>>(p);
}
```

```cpp
#include <hip/hip_runtime.h>
#include <rccl/rccl.h>
#include <cstddef>
#include <cstdio>
#include <type_traits>

typedef float f32x4 __attribute__((ext_vector_type(4)));
typedef float f32x16 __attribute__((ext_vector_type(16)));
typedef _Float16 f16x8 __attribute__((ext_vector_type(8)));
typedef _Float16 f16x2 __attribute__((ext_vector_type(2)));
typedef float f32x2 __attribute__((ext_vector_type(2)));
typedef unsigned int u32x4 __attribute__((ext_vector_type(4)));
typedef unsigned int u32x2 __attribute__((ext_vector_type(2)));
typedef unsigned int u32x8 __attribute__((ext_vector_type(8)));

constexpr int NR = 8;
constexpr int ML = 512;
constexpr int KK = 4096, NN = 4096;
constexpr int NS = 512;
constexpr int ROUNDS = 8;
constexpr int GRID = 256;
constexpr unsigned long long DEADLINE = 50000000ull;
constexpr unsigned OOB = 0xFFFF0000u;

constexpr size_t WS_DONE = 0;
constexpr size_t WS_TMO = 1024;
constexpr size_t WS_CNT = 4096;
constexpr size_t WS_SLAB = 1u << 20;
constexpr size_t SLAB_BYTES = (size_t)16 * 4 * 8 * 4 * 4 * 3 * 1024;
constexpr size_t WS_NEED = WS_SLAB + SLAB_BYTES;

constexpr int RING_BYTES = 65536;
constexpr int LDS_DONE = 2 * RING_BYTES;
constexpr int LDS_BYTES = LDS_DONE + 16;

constexpr int B_LOOK = 8;
constexpr int A_POS = 32;
constexpr int F_POS = 55;

constexpr int AUX_SC1 = 16;
constexpr int AUX_SYS = 17;

struct Params {
  const float* x;
  const float* w;
  char* myws;
  int rank, pad;
  float* out[NR];
  char* ws[NR];
};
static_assert(offsetof(Params, x) == 0 && offsetof(Params, w) == 8 && offsetof(Params, myws) == 16 && offsetof(Params, rank) == 24,
              "the kernel reads the first 32 bytes of its arguments in one piece");

__device__ __forceinline__ float silu(float y) {
  const float c = fminf(fmaxf(y, -60.f), 60.f);
  return y * __builtin_amdgcn_rcpf(1.f + __expf(-c));
}

__device__ __forceinline__ unsigned pack2(float a, float b) {
  const f32x2 v = {a, b};
  return __builtin_bit_cast(unsigned, __builtin_convertvector(v, f16x2));
}

__device__ __forceinline__ void strip_map(int j, int u, int rank, int& dest, int& dcol) {
  int di, sd;
  if (j < 7) {
    if (u == j) { di = 7; sd = j; }
    else {
      const int t = u - (u > j);
      if (t < 14) { const int h = t >= 7; di = t - 7 * h; sd = 2 * j + h; }
      else { di = j; sd = 14; }
    }
  } else {
    if (u < 7) { di = u; sd = 15; }
    else { di = 7; sd = u; }
  }
  dest = (rank + 1 + di) & 7;
  dcol = sd * 32;
}

__device__ __forceinline__ int bslot(int c, int i) {
  return ((c + 2 * i) & 7) + 8 * (i & 1) + 16 * (__builtin_popcount(c) & 1);
}

#define SB() __builtin_amdgcn_sched_barrier(0)
template <int V> using IC = std::integral_constant<int, V>;

__global__ void __launch_bounds__(256, 1) fused(Params p) {
  __shared__ __attribute__((aligned(16))) char smem[LDS_BYTES];
  const int tid = threadIdx.x, lane = tid & 63;
  const int w = __builtin_amdgcn_readfirstlane(tid >> 6);
  const u32x8 hdr = *(const __attribute__((address_space(4))) u32x8*)__builtin_amdgcn_kernarg_segment_ptr();
  const float* const px = (const float*)(((unsigned long long)hdr[1] << 32) | hdr[0]);
  const float* const pw = (const float*)(((unsigned long long)hdr[3] << 32) | hdr[2]);
  char* const myws = (char*)(__attribute__((address_space(1))) char*)(((unsigned long long)hdr[5] << 32) | hdr[4]);
  const int rank = (int)hdr[6];
  const int bid = blockIdx.x;
  const int g8 = bid & 7, idx = bid >> 3;
  const int nh = g8 & 1, kq = g8 >> 1;
  const int mi = idx & 3, ng = idx >> 2;
  const int slot = nh * 8 + ng;
  const int rot = (5 * mi + 8 * (ng >> 2)) & 15;
  const int qk = (kq - mi) & 3;

  const __amdgpu_buffer_rsrc_t rx =
      __builtin_amdgcn_make_buffer_rsrc((void*)px, 0, (int)((unsigned)ML * KK * 4u), 0x00020000);
  f32x4 t[2][8];
  auto xi = [&](int q) {
    const int ac = ((q >> 2) * 4 + (lane >> 4) + rot) & 15;
    const unsigned vo = (unsigned)(kq * 1024 + ac * 64 + (lane & 15) * 4) * 4u;
#pragma unroll
    for (int r = 0; r < 8; ++r)
      t[q & 1][r] = __builtin_bit_cast(f32x4, __builtin_amdgcn_raw_buffer_load_b128(
                                                  rx, vo, (unsigned)(mi * 128 + w * 32 + ((q + ng) & 3) * 8 + r) * (KK * 4u), 0));
    SB();
  };
  xi(0); xi(1);
  if (tid == 0) *(unsigned*)(smem + LDS_DONE) = 0u;

  const __amdgpu_buffer_rsrc_t rw =
      __builtin_amdgcn_make_buffer_rsrc((void*)pw, 0, (int)((unsigned)KK * NN * 4u), 0x00020000);
  const unsigned voff = (unsigned)(8 * (lane >> 3)) * (NN * 4u) + (unsigned)(lane & 7) * 16u;
  unsigned rowoff[4];
#pragma unroll
  for (int c = 0; c < 4; ++c) rowoff[c] = (unsigned)(kq * 1024 + ((4 * w + c + rot) & 15) * 64) * (NN * 4u);
  int tdest, tdcol;
  strip_map(lane & 7, slot, rank, tdest, tdcol);
  const unsigned coltab = (unsigned)(tdest * NS + tdcol) * 4u;
  unsigned wroff[4];
#pragma unroll
  for (int i = 0; i < 4; ++i)
    wroff[i] = (unsigned)(w * 16384 + (lane >> 4) * 1024 + (((lane >> 3) & 1) * 32 + bslot(lane & 7, i)) * 16);
  const unsigned rdoff = (unsigned)(((lane >> 5) * 32 + bslot((lane & 31) >> 2, lane & 3)) * 16);
  f32x4 buf[2][8];
  u32x4 pk[4];

  auto w_issue1 = [&](unsigned base, int c, int jj) {
    buf[c & 1][jj] = __builtin_bit_cast(f32x4, __builtin_amdgcn_raw_buffer_load_b128(rw, voff, base + (unsigned)jj * (NN * 4u), 0));
  };
  auto w_issue = [&](int j, int c) {
    const unsigned base = rowoff[c] + (unsigned)__builtin_amdgcn_readlane(coltab, j);
#pragma unroll
    for (int jj = 0; jj < 8; ++jj) w_issue1(base, c, jj);
    SB();
  };

  w_issue(0, 0); w_issue(0, 1);
  unsigned long long optr = (unsigned long long)p.out[tdest & 7];
  unsigned outlo = 0, outhi = 0;
  SB();

  const __amdgpu_buffer_rsrc_t rs =
      __builtin_amdgcn_make_buffer_rsrc(myws + WS_SLAB, 0, (int)SLAB_BYTES, 0x00020000);
  const __amdgpu_buffer_rsrc_t rc = __builtin_amdgcn_make_buffer_rsrc(myws, 0, 4096, 0x00020000);
  unsigned* const tmo = (unsigned*)(myws + WS_TMO);
  unsigned call = 0, tag = 0;
  f32x4 pend0 = {0.f, 0.f, 0.f, 0.f}, pend1 = pend0;
  u32x4 pr[3];
  unsigned tm = 0;
  int lo = 0;

  auto tile_off = [&](int j, int q) { return (unsigned)((((slot * 4 + mi) * 8 + j) * 4 + w) * 4 + q) * 3u * 1024u; };
  auto gather1 = [&](unsigned o, int i) { pr[i] = __builtin_amdgcn_raw_buffer_load_b128(rs, o, (unsigned)i * 1024u, AUX_SC1); };
  auto gather_tm = [&]() { tm = __builtin_amdgcn_raw_buffer_load_b32(rc, (unsigned)WS_TMO, 0, AUX_SC1); };
  auto gather_off = [&]() { return tile_off(lo, qk) + (unsigned)lane * 16u; };
  auto gather_all = [&]() {
    const unsigned o = gather_off();
#pragma unroll
    for (int i = 0; i < 3; ++i) gather1(o, i);
    gather_tm();
  };
  auto tags_ok = [&]() -> bool {
    asm volatile("" : "+v"(tm));
    unsigned all1 = ~0u, any1 = 0u;
#pragma unroll
    for (int i = 0; i < 3; ++i)
#pragma unroll
      for (int e = 0; e < 4; ++e) { all1 &= pr[i][e]; any1 |= pr[i][e]; }
    const bool ok = tag ? (all1 & 1u) : !(any1 & 1u);
    return __all(ok || tm == call);
  };
  auto gather_finish = [&](f32x4& y, bool must) -> bool {
    if (!tags_ok()) {
      if (!must) return false;
      const unsigned long long t0 = __builtin_amdgcn_s_memrealtime();
#pragma unroll 1
      do {
        if (__builtin_amdgcn_s_memrealtime() - t0 > DEADLINE) {
          __hip_atomic_store(tmo, call, __ATOMIC_RELAXED, __HIP_MEMORY_SCOPE_AGENT);
          tm = call;
          break;
        }
        __builtin_amdgcn_s_sleep(1);
        gather_all();
      } while (!tags_ok());
    }
    const float poison = __any(tm == call) ? __builtin_nanf("") : 0.f;
    const f32x4 own = (lo & 1) ? pend1 : pend0;
    const f32x4 p0 = __builtin_bit_cast(f32x4, pr[0]), p1 = __builtin_bit_cast(f32x4, pr[1]), p2 = __builtin_bit_cast(f32x4, pr[2]);
#pragma unroll
    for (int e = 0; e < 4; ++e) y[e] = silu(own[e] + p0[e] + p1[e] + p2[e]) + poison;
    return true;
  };
  const unsigned ovoff = (unsigned)((w * 32 + 8 * qk + 4 * (lane >> 5)) * NS + (lane & 31)) * 4u;
  auto out_rsrc = [&](int j) {
    const unsigned hi = (unsigned)__builtin_amdgcn_readlane(outhi, j), lo32 = (unsigned)__builtin_amdgcn_readlane(outlo, j);
    const unsigned long long a = ((unsigned long long)hi << 32) | (unsigned long long)lo32;
    return __builtin_amdgcn_make_buffer_rsrc((void*)a, 0, 128 * NS * 4, 0x00020000);
  };
  auto out1 = [&](const f32x4& y, __amdgpu_buffer_rsrc_t r, unsigned o, int e) {
    const u32x4 u = __builtin_bit_cast(u32x4, y);
    __builtin_amdgcn_raw_buffer_store_b32(u[e], r, o, (unsigned)e * (NS * 4u), AUX_SYS);
  };
  auto part_out = [&](int j, const f32x16& acc) {
#pragma unroll
    for (int q = 0; q < 4; ++q) {
      const f32x4 a = {acc[4 * q], acc[4 * q + 1], acc[4 * q + 2], acc[4 * q + 3]};
      if (q == qk) { if (j & 1) pend1 = a; else pend0 = a; }
      u32x4 v = __builtin_bit_cast(u32x4, a);
#pragma unroll
      for (int i = 0; i < 4; ++i) v[i] = (v[i] & ~1u) | tag;
      const unsigned off = q == qk ? OOB : tile_off(j, q) + (unsigned)((kq - mi - q - 1) & 3) * 1024u + (unsigned)lane * 16u;
      __builtin_amdgcn_raw_buffer_store_b128(v, rs, off, 0, AUX_SC1);
    }
    SB();
  };

  f16x8 A[64];
  unsigned touch = 0;
  {
    char* stage = smem + RING_BYTES + w * 16384;
    auto xc = [&](int q) {
      const int ch = lane >> 1;
#pragma unroll
      for (int r = 0; r < 8; ++r) {
        const int row = ((q + ng) & 3) * 8 + r;
        const u32x2 v = {pack2(t[q & 1][r][0], t[q & 1][r][1]), pack2(t[q & 1][r][2], t[q & 1][r][3])};
        *(u32x2*)(stage + row * 512 + (((ch & 16) | ((ch ^ row) & 15)) << 4) + (lane & 1) * 8) = v;
      }
      SB();
    };
    auto xr = [&](int b) {
      const int r = lane & 31;
      f16x8 v[16];
#pragma unroll
      for (int e = 0; e < 16; ++e) {
        const int ch = 2 * e + (lane >> 5);
        v[e] = *(const f16x8*)(stage + r * 512 + (((ch & 16) | ((ch ^ r) & 15)) << 4));
      }
      SB();
#pragma unroll
      for (int e = 0; e < 16; ++e) asm volatile("" : "=a"(A[b * 16 + e]) : "0"(v[e]));
      SB();
    };
    unsigned done0 = 0;
#pragma unroll
    for (int q = 0; q < 16; ++q) {
      xc(q);
      if (q + 2 < 16) xi(q + 2);
      if (q == 6) done0 = __builtin_amdgcn_raw_buffer_load_b32(rc, (unsigned)(WS_DONE + g8 * 128), 0, AUX_SC1);
      if (q == 8) {
        asm volatile("" : "+v"(optr));
        const unsigned long long a = optr + ((size_t)(rank * ML + mi * 128) * NS + tdcol) * 4u;
        outlo = (unsigned)a;
        outhi = (unsigned)(a >> 32);
      }
      if ((q & 3) == 3) xr(q >> 2);
      if (q >= 4 && q < 12 && (q & 1) == 0) {
        const int c = (q - 4) >> 1;
#pragma unroll
        for (int i = 0; i < 4; ++i) {
#pragma unroll
          for (int h = 0; h < 4; ++h) pk[i][h] = pack2(buf[c & 1][2 * h][i], buf[c & 1][2 * h + 1][i]);
          asm volatile("" : "+v"(pk[i]));
        }
        if (c + 2 < 4) w_issue(0, c + 2);
#pragma unroll
        for (int i = 0; i < 4; ++i) *(u32x4*)(smem + c * 4096 + wroff[i]) = pk[i];
        SB();
      }
      if (q >= 14) w_issue(1, q - 14);
    }
    if (w == 0) {
      const unsigned long long a = lane < 8 ? (((unsigned long long)outhi << 32) | outlo)
                                            : (unsigned long long)(myws + WS_SLAB + tile_off(0, 0));
      if (lane < 9)
        touch = __hip_atomic_load((const __attribute__((address_space(1))) unsigned*)a, __ATOMIC_RELAXED, __HIP_MEMORY_SCOPE_SYSTEM);
    }
    asm volatile("" : "+v"(done0));
    call = __builtin_amdgcn_readfirstlane(done0 / (GRID / 8) + 1u);
    tag = call & 1u;
  }
  asm volatile("s_waitcnt lgkmcnt(0)" ::: "memory");
  __builtin_amdgcn_s_barrier();

  auto round_body = [&](int j, auto next_tag, auto next2_tag) {
    constexpr bool NEXT = decltype(next_tag)::value, NEXT2 = decltype(next2_tag)::value;
    const char* ring = smem + (j & 1) * RING_BYTES + rdoff;
    char* ringn = smem + ((j + 1) & 1) * RING_BYTES;
    const unsigned col1 = (unsigned)__builtin_amdgcn_readlane(coltab, (j + 1) & 7);
    const unsigned col2 = (unsigned)__builtin_amdgcn_readlane(coltab, (j + 2) & 7);
    const bool have = lo < j;
    const unsigned go = have ? gather_off() : OOB;
    f32x16 acc;
    f32x4 y = {0.f, 0.f, 0.f, 0.f};
    __amdgpu_buffer_rsrc_t ro = rs;
    unsigned oo = OOB;
    f16x8 B[B_LOOK];
#pragma unroll
    for (int e = 0; e < B_LOOK; ++e) B[e] = *(const f16x8*)(ring + e * 1024);
    SB();
    auto seg = [&](auto lo_tag, auto hi_tag) {
      constexpr int LO = decltype(lo_tag)::value, HI = decltype(hi_tag)::value;
#pragma unroll
      for (int f = LO; f < HI; ++f) {
        const int c = f >> 4, g = f & 15;
        if (f == 0) asm volatile("v_mfma_f32_32x32x16_f16 %0, %1, %2, 0" : "=&v"(acc) : "a"(A[f]), "v"(B[f % B_LOOK]));
        else asm volatile("v_mfma_f32_32x32x16_f16 %0, %1, %2, %0" : "+v"(acc) : "a"(A[f]), "v"(B[f % B_LOOK]));
        if (f + B_LOOK < 64) B[f % B_LOOK] = *(const f16x8*)(ring + (f + B_LOOK) * 1024);
        if (NEXT) {
          if (c > 0 && g < 4) *(u32x4*)(ringn + (c - 1) * 4096 + wroff[g]) = pk[g];
          const int h = g >> 2, i = g & 3;
          pk[i][h] = pack2(buf[c & 1][2 * h][i], buf[c & 1][2 * h + 1][i]);
          asm volatile("" : "+v"(pk[i]));
          if (i == 3) {
            if (c < 2) {
              w_issue1(rowoff[c + 2] + col1, c, 2 * h);
              w_issue1(rowoff[c + 2] + col1, c, 2 * h + 1);
            } else if (NEXT2) {
              w_issue1(rowoff[c - 2] + col2, c, 2 * h);
              w_issue1(rowoff[c - 2] + col2, c, 2 * h + 1);
            }
          }
        }
        if (f >= A_POS && f < A_POS + 3) gather1(go, f - A_POS);
        if (f == A_POS + 4) gather_tm();
        if (f == F_POS + 1) out1(y, ro, oo, 0);
        if (f == F_POS + 2) out1(y, ro, oo, 1);
        if (f == F_POS + 3) out1(y, ro, oo, 2);
        if (f == F_POS + 5) out1(y, ro, oo, 3);
        SB();
      }
    };
    seg(IC<0>{}, IC<F_POS + 1>{});
    ro = out_rsrc(lo & 7);
    if (have && gather_finish(y, j - lo >= 2)) { oo = ovoff; ++lo; }
    SB();
    seg(IC<F_POS + 1>{}, IC<64>{});
    if (NEXT) {
#pragma unroll
      for (int i = 0; i < 4; ++i) *(u32x4*)(ringn + 3 * 4096 + wroff[i]) = pk[i];
      SB();
    }
    asm volatile("s_nop 11" : "+v"(acc));
    part_out(j, acc);
    if (NEXT) {
      asm volatile("s_waitcnt lgkmcnt(0)" ::: "memory");
      __builtin_amdgcn_s_barrier();
    }
  };
#pragma unroll 1
  for (int j = 0; j < ROUNDS - 2; ++j) round_body(j, std::true_type{}, std::true_type{});
  round_body(ROUNDS - 2, std::true_type{}, std::false_type{});
  round_body(ROUNDS - 1, std::false_type{}, std::false_type{});

  {
    f32x4 y;
    if (lo < ROUNDS - 1) {
      gather_all();
      gather_finish(y, true);
      const __amdgpu_buffer_rsrc_t r6 = out_rsrc(lo);
#pragma unroll
      for (int e = 0; e < 4; ++e) out1(y, r6, ovoff, e);
      ++lo;
    }
    __builtin_amdgcn_s_sleep(10);
    gather_all();
    gather_finish(y, true);
    const __amdgpu_buffer_rsrc_t r = out_rsrc(ROUNDS - 1);
#pragma unroll
    for (int e = 0; e < 4; ++e) out1(y, r, ovoff, e);
    asm volatile("s_waitcnt vmcnt(0)" ::: "memory");
    unsigned n = 0;
    if (lane == 0)
      n = __hip_atomic_fetch_add((unsigned*)(smem + LDS_DONE), 1u, __ATOMIC_RELAXED, __HIP_MEMORY_SCOPE_WORKGROUP);
    n = __builtin_amdgcn_readfirstlane(n);
    if (n == 3u && lane < NR) {
      if (lane != rank)
        __hip_atomic_fetch_add((unsigned*)(p.ws[lane] + WS_CNT + (rank * 8 + g8) * 128), 1u, __ATOMIC_RELAXED,
                               __HIP_MEMORY_SCOPE_SYSTEM);
      else
        __hip_atomic_fetch_add((unsigned*)(myws + WS_DONE + g8 * 128), 1u, __ATOMIC_RELAXED, __HIP_MEMORY_SCOPE_AGENT);
    }
  }
  asm volatile("" :: "v"(touch));

  if (bid == 0 && w == 0) {
    const int src = lane >> 3;
    bool bad = false;
    unsigned* f = (unsigned*)(myws + WS_CNT + lane * 128);
    const unsigned need = call * (GRID / 8);
    const unsigned long long t0 = __builtin_amdgcn_s_memrealtime();
#pragma unroll 1
    for (unsigned spin = 1;; ++spin) {
      const unsigned v = __hip_atomic_load(f, __ATOMIC_RELAXED, __HIP_MEMORY_SCOPE_SYSTEM);
      if (__all(src == rank || v >= need)) break;
      if ((spin & 7u) == 0u) {
        if (__hip_atomic_load(tmo, __ATOMIC_RELAXED, __HIP_MEMORY_SCOPE_AGENT) == call ||
            __builtin_amdgcn_s_memrealtime() - t0 > DEADLINE) {
          __hip_atomic_store(tmo, call, __ATOMIC_RELAXED, __HIP_MEMORY_SCOPE_AGENT);
          bad = true;
          break;
        }
      }
      __builtin_amdgcn_s_sleep(1);
    }
    __builtin_amdgcn_fence(__ATOMIC_ACQUIRE, "");
    if (bad) {
      asm volatile("s_waitcnt vmcnt(0)" ::: "memory");
      if (lane == 0) p.out[rank][(size_t)rank * ML * NS] = __builtin_nanf("");
    }
  }
}

extern "C" void kernel_launch(int rank, int nranks, ncclComm_t comm, hipStream_t stream,
                              void* const* d_in, const int* in_sizes, int n_in,
                              void* d_out, int out_size, const int* in_dtypes, int out_dtype,
                              void* const* d_out_all, void* const* d_ws_all, size_t ws_bytes) {
  if (nranks != NR || n_in < 2 || in_sizes[0] != ML * KK || in_sizes[1] != KK * NN || out_size != NR * ML * NS ||
      ws_bytes < WS_NEED || (unsigned)rank >= (unsigned)NR) {
    fprintf(stderr, "kernel_launch: unexpected problem shape\n");
    return;
  }
  Params p;
  p.x = (const float*)d_in[0];
  p.w = (const float*)d_in[1];
  for (int i = 0; i < NR; ++i) {
    p.out[i] = (float*)d_out_all[i];
    p.ws[i] = (char*)d_ws_all[i];
  }
  p.myws = (char*)d_ws_all[rank];
  p.rank = rank;
  fused<<<GRID, 256, 0, stream>>>(p);
}
```
